```python
import numpy as np

kernel_name = "gpu_gemm_a2a_m4096_k4096_n16384_bf16_silu"


ABSMAX_RTOL = 0.008

COMPUTE_DTYPE = "bfloat16"
EPILOGUE = "silu"
IN_QUANT = None
OUT_QUANT = None
IN_DTYPES = {"x": "bfloat16", "w_mat": "bfloat16"}
WORLD_SIZE = 8

def _to_bf16(x):
    u = np.ascontiguousarray(x, dtype=np.float32).view(np.uint32)
    return ((u + 0x7FFF + ((u >> 16) & 1)) & 0xFFFF0000).view(np.float32)

def setup_inputs():

    x = _to_bf16(np.random.randn(4096, 4096).astype(np.float32))
    w_mat = _to_bf16((np.random.randn(4096, 16384) / np.sqrt(4096)).astype(np.float32))
    return {"x": x, "w_mat": w_mat}

def _epilogue(y):
    return (y / (1.0 + np.exp(-np.clip(y, -60.0, 60.0)))).astype(np.float32)

def reference(x, w_mat):
    y = (x @ w_mat).astype(np.float32)
    return _epilogue(y)
```

```cpp
#include <hip/hip_runtime.h>
#include <rccl/rccl.h>
#include <cstdio>

#define HIP_CHECK(expr) do { hipError_t _e = (expr); if (_e != hipSuccess) { \
    fprintf(stderr, "HIP error %s at %s:%d\n", hipGetErrorString(_e), __FILE__, __LINE__); \
    return; } } while (0)

namespace {
#define LAS __attribute__((address_space(3)))
typedef short bf16x8 __attribute__((ext_vector_type(8)));
typedef short s16x4 __attribute__((ext_vector_type(4)));
typedef float f32x4 __attribute__((ext_vector_type(4)));
typedef unsigned u32x4 __attribute__((ext_vector_type(4)));
typedef unsigned u32x2 __attribute__((ext_vector_type(2)));

constexpr int MLOC = 512, KD = 4096, ND = 16384, NR = 8, NPR = 2048;
#define NSTAGE_ 5
#define SEGX_ 7
#define SEGY_ 7
#define AUX_ 17
#define REL() do {} while (0)
#define SYNCT_ 4
constexpr int BM = 128, BN = 128, BK = 64, S = NSTAGE_;
constexpr int NKT = KD / BK;
constexpr int A_BYTES = BM * BK * 2, B_BYTES = BK * BN * 2, STAGE_BYTES = A_BYTES + B_BYTES;
constexpr int LDS_BYTES = S * STAGE_BYTES;
constexpr int NWG = 256;
constexpr int UPW = 2;
constexpr int WM = BM / 2, WN = BN / 2;
constexpr int MI = WM / 16, NI = WN / 16;
constexpr int NLD = 8;
constexpr int NTHREADS = 8 * 64;
constexpr int BROW = BN * 2;
constexpr int G = UPW * NKT;
constexpr unsigned long long DEADLINE = 50000000ull;

constexpr int WS_TMO = 64;
constexpr int WS_TICKET = 1024;
constexpr int WS_XFLAG = 2048;
constexpr int WS_YFLAG = 4096;
constexpr size_t WS_XSTAGE = 1u << 20;
constexpr size_t XBLK_BYTES = (size_t)128 * KD * 2;

struct Params {
    const unsigned short* x; const unsigned short* w;
    void* out_all[8]; void* ws_all[8];
    int rank; int single;
};

__device__ __forceinline__ void bar() { asm volatile("" ::: "memory"); __builtin_amdgcn_s_barrier(); asm volatile("" ::: "memory"); }
__device__ __forceinline__ unsigned cvt_pk_bf16(float lo, float hi) { unsigned r; asm("v_cvt_pk_bf16_f32 %0, %1, %2" : "=v"(r) : "v"(lo), "v"(hi)); return r; }
__device__ __forceinline__ float silu(float y) {
    float c = __builtin_amdgcn_fmed3f(y, -60.f, 60.f);
    float e = __builtin_amdgcn_exp2f(c * -1.44269504088896340736f);
    return y * __builtin_amdgcn_rcpf(1.0f + e);
}

__device__ __forceinline__ void unit_coords(int wg, int ui, int rank, int& src, int& rb, int& dst, int& cb) {
    const int xcd = wg & 7, idx = wg >> 3;
    int u;
    if (ui == 0) u = xcd * 32 + idx;
    else if (idx < 10) u = 256 + xcd * 10 + idx;
    else u = -1;
    if (u >= 0) { const int seq = u / 3; rb = u - seq * 3; src = rank; dst = (rank + 1 + seq % 7) & 7; cb = seq / 7; }
    else {
        const int v = idx - 10;
        const int c = v / 11, r = v - c * 11;
        dst = rank; cb = xcd * 2 + c;
        if (r < 4) { src = rank; rb = r; } else { src = (rank + r - 3) & 7; rb = 3; }
    }
}

#define TS(i) do {} while (0)

__global__ void __launch_bounds__(NTHREADS, 2) fused_gemm_a2a(Params p) {
    extern __shared__ __attribute__((aligned(16))) unsigned char smem[];
    LAS unsigned char* lds = (LAS unsigned char*)smem;
    const int tid = threadIdx.x, wid = __builtin_amdgcn_readfirstlane(tid >> 6), lane = tid & 63;
    const int wg = blockIdx.x;
    unsigned* ws = (unsigned*)p.ws_all[p.rank];
    TS(0);

    if (wid < 4) {
        const int wr = wid >> 1, wc = wid & 1;
        const int fr = lane & 15, fq = lane >> 4;
        const unsigned ticket = __hip_atomic_load(ws + WS_TICKET + wg, __ATOMIC_RELAXED, __HIP_MEMORY_SCOPE_AGENT);
        const unsigned xo = (unsigned)((wg * 4 + wid) * 1024 + lane * 16);
        const u32x4 xv = *(const u32x4*)((const char*)p.x + (size_t)3 * XBLK_BYTES + xo);
        const unsigned long long tws = (unsigned long long)p.ws_all[lane & 7], tout = (unsigned long long)p.out_all[lane & 7];
        auto tab = [&](unsigned long long t, int r) {
            const unsigned lo = __builtin_amdgcn_readlane((unsigned)t, r), hi = __builtin_amdgcn_readlane((unsigned)(t >> 32), r);
            return (char*)(((unsigned long long)hi << 32) | lo);
        };
        const unsigned a0 = (unsigned)((wr * WM + fr) * 128 + ((fq ^ (fr >> 1)) * 16));
        const unsigned a1 = a0 ^ 64u;
        const int q = (lane & 15) >> 2, pp = lane & 3, h = q | ((fq & 1) << 2);
        unsigned bo[NI];
#pragma unroll
        for (int ni = 0; ni < NI; ++ni) {
            const int n16 = wc * NI + ni;
            bo[ni] = (unsigned)(A_BYTES + (8 * fq + q) * BROW + ((n16 ^ h) * 32) + (pp >> 1) * 16 + (pp & 1) * 8);
        }
        f32x4 acc[MI][NI];
#pragma unroll
        for (int mi = 0; mi < MI; ++mi)
#pragma unroll
            for (int ni = 0; ni < NI; ++ni) acc[mi][ni] = (f32x4){0.f, 0.f, 0.f, 0.f};
        bf16x8 af[2][MI], bf[2][NI];
        u32x4 pk[8];
#pragma unroll
        for (int i = 0; i < 8; ++i) pk[i] = (u32x4){0u, 0u, 0u, 0u};
#define LOADF(set, sbase, ks) do { \
        _Pragma("unroll") for (int mi = 0; mi < MI; ++mi) af[set][mi] = *(const LAS bf16x8*)(lds + (sbase) + ((ks) ? a1 : a0) + mi * 2048); \
        _Pragma("unroll") for (int ni = 0; ni < NI; ++ni) { \
            s16x4 lo = __builtin_amdgcn_ds_read_tr16_b64_v4i16((LAS s16x4*)(lds + (sbase) + bo[ni] + (ks) * 32 * BROW)); \
            s16x4 hi = __builtin_amdgcn_ds_read_tr16_b64_v4i16((LAS s16x4*)(lds + (sbase) + bo[ni] + (ks) * 32 * BROW + 4 * BROW)); \
            bf[set][ni] = __builtin_shufflevector(lo, hi, 0, 1, 2, 3, 4, 5, 6, 7); } } while (0)
#define MMA(set) do { \
        _Pragma("unroll") for (int mi = 0; mi < MI; ++mi) _Pragma("unroll") for (int ni = 0; ni < NI; ++ni) \
            acc[mi][ni] = __builtin_amdgcn_mfma_f32_16x16x32_bf16(bf[set][ni], af[set][mi], acc[mi][ni], 0, 0, 0); } while (0)
#define INTERLEAVE() do { \
        _Pragma("unroll") for (int i = 0; i < MI + 2 * NI; ++i) { __builtin_amdgcn_sched_group_barrier(0x8, 1, 0); __builtin_amdgcn_sched_group_barrier(0x100, 1, 0); } \
        __builtin_amdgcn_sched_group_barrier(0x8, MI * NI - (MI + 2 * NI), 0); } while (0)
#define TILE() do { \
            const unsigned sn = (sb + STAGE_BYTES == S * STAGE_BYTES) ? 0u : sb + STAGE_BYTES; \
            LOADF(1, sb, 1); MMA(0); INTERLEAVE(); \
            LOADF(0, sn, 0); MMA(1); INTERLEAVE(); \
            sb = sn; } while (0)
        bar();
        TS(1);
        LOADF(0, 0u, 0);
        unsigned sb = 0;
        unsigned call = 0, tmo = 0;
        __amdgpu_buffer_rsrc_t rs = __builtin_amdgcn_make_buffer_rsrc((void*)p.x, 0, 0, 0x00020000);
        unsigned gofs = 0; int pdst = p.rank, pflag = 0;
#pragma unroll 1
        for (int ui = 0; ui < UPW; ++ui) {
            const int segt = ui == 0 ? SEGX_ : SEGY_;
            const int tsel = segt - 1;
#pragma unroll
            for (int seg = 0; seg < 10; ++seg) {
                const int nt = seg < 8 ? ((ui == 0 && seg == 7) ? 0 : segt) : seg == 8 ? SYNCT_ : NKT - SYNCT_ - (ui == 0 ? 7 * SEGX_ : 8 * SEGY_);
                const int ts_ = seg < 8 ? tsel : nt - 1;
#pragma unroll 1
                for (int t = 0; t < nt; ++t) {
                    TILE();
                    if (seg < 9 && t == ts_) {
                        if (ui == 0) {
                            if (seg == 0) {
                                call = (unsigned)__builtin_amdgcn_readfirstlane(ticket) + 1u;
                                if (tid == 0) __hip_atomic_store(ws + WS_TICKET + wg, call, __ATOMIC_RELAXED, __HIP_MEMORY_SCOPE_AGENT);
                            }
                            if (!p.single) {
                                if (seg < 7) {
                                    const int pr = (p.rank + 1 + (seg + wg + wid) % 7) & 7;
                                    const __amdgpu_buffer_rsrc_t xr = __builtin_amdgcn_make_buffer_rsrc(tab(tws, pr) + WS_XSTAGE + ((size_t)(call & 1) * NR + p.rank) * XBLK_BYTES, 0, (int)XBLK_BYTES, 0x00020000);
                                    __builtin_amdgcn_raw_buffer_store_b128(xv, xr, xo, 0, AUX_);
                                } else if (seg == 8) {
                                    asm volatile("s_waitcnt vmcnt(0)" ::: "memory");
                                }
                            }
                        } else if (seg < 8) {
                            __builtin_amdgcn_raw_buffer_store_b128(pk[seg], rs, gofs + (unsigned)(((seg >> 1) * 16 * NPR + (seg & 1) * 32) * 2), 0, AUX_);
                        } else {
                            if (pdst != p.rank) {
                                asm volatile("s_waitcnt vmcnt(0)" ::: "memory"); REL();
                                if (lane == pdst) __hip_atomic_store((unsigned*)tws + WS_YFLAG + 256 * p.rank + pflag, call, __ATOMIC_RELAXED, __HIP_MEMORY_SCOPE_SYSTEM);
                            }
                            tmo = __hip_atomic_load(ws + WS_TMO, __ATOMIC_RELAXED, __HIP_MEMORY_SCOPE_AGENT);
                        }
                    }
                    bar();
                }
                if (seg == 8 && ui == 0 && !p.single && wid == 0) {
                    REL();
                    if (lane < 8 && lane != p.rank) __hip_atomic_store((unsigned*)tws + WS_XFLAG + 256 * p.rank + wg, call, __ATOMIC_RELAXED, __HIP_MEMORY_SCOPE_SYSTEM);
                    TS(9);
                }
            }
            TS(2 + 2 * ui);
            int src, rb, dst, cb; unit_coords(wg, ui, p.rank, src, rb, dst, cb);
            dst = __builtin_amdgcn_readfirstlane(dst) & 7; src = __builtin_amdgcn_readfirstlane(src) & 7;
            if (p.single) dst = p.rank;
            pdst = dst; pflag = (cb * 3 + rb) * 4 + wid;
            rs = __builtin_amdgcn_make_buffer_rsrc(tab(tout, dst), 0, 4096 * NPR * 2, 0x00020000);
            gofs = (unsigned)(((src * MLOC + rb * BM + wr * WM + fr) * NPR + cb * BN + wc * WN + (fq & 1) * 16 + (fq >> 1) * 8) * 2);
            const bool bad = src != p.rank && (unsigned)__builtin_amdgcn_readfirstlane(tmo) == call;
#pragma unroll
            for (int mi = 0; mi < MI; ++mi)
#pragma unroll
                for (int pr = 0; pr < 2; ++pr) {
                    const f32x4 va = acc[mi][2 * pr], vb = acc[mi][2 * pr + 1];
                    const unsigned ax = cvt_pk_bf16(silu(va[0]), silu(va[1])), ay = cvt_pk_bf16(silu(va[2]), silu(va[3]));
                    const unsigned bx = cvt_pk_bf16(silu(vb[0]), silu(vb[1])), by = cvt_pk_bf16(silu(vb[2]), silu(vb[3]));
                    const auto sx = __builtin_amdgcn_permlane16_swap(ax, bx, false, false);
                    const auto sy = __builtin_amdgcn_permlane16_swap(ay, by, false, false);
                    u32x4 o = {sx[0], sy[0], sx[1], sy[1]};
                    if (bad) o = (u32x4){0x7fc07fc0u, 0x7fc07fc0u, 0x7fc07fc0u, 0x7fc07fc0u};
                    pk[mi * 2 + pr] = o;
                    acc[mi][2 * pr] = (f32x4){0.f, 0.f, 0.f, 0.f}; acc[mi][2 * pr + 1] = (f32x4){0.f, 0.f, 0.f, 0.f};
                }
            TS(3 + 2 * ui);
        }
#pragma unroll
        for (int i = 0; i < 8; ++i) __builtin_amdgcn_raw_buffer_store_b128(pk[i], rs, gofs + (unsigned)(((i >> 1) * 16 * NPR + (i & 1) * 32) * 2), 0, AUX_);
        if (pdst != p.rank) {
            asm volatile("s_waitcnt vmcnt(0)" ::: "memory"); REL();
            if (lane == pdst) __hip_atomic_store((unsigned*)tws + WS_YFLAG + 256 * p.rank + pflag, call, __ATOMIC_RELAXED, __HIP_MEMORY_SCOPE_SYSTEM);
        }
        TS(14);
        if (wg == 0 && wid == 0 && !p.single) {
            const unsigned long long t0 = __builtin_amdgcn_s_memrealtime();
            bool to = false;
            for (int i = 1; i < NR && !to; ++i) {
                const unsigned* fp = ws + WS_YFLAG + 256 * ((p.rank + i) & 7) + lane * 3;
                for (;;) {
                    const unsigned f0 = __hip_atomic_load(fp, __ATOMIC_RELAXED, __HIP_MEMORY_SCOPE_SYSTEM), f1 = __hip_atomic_load(fp + 1, __ATOMIC_RELAXED, __HIP_MEMORY_SCOPE_SYSTEM),
                                   f2 = __hip_atomic_load(fp + 2, __ATOMIC_RELAXED, __HIP_MEMORY_SCOPE_SYSTEM);
                    if (__all(f0 >= call && f1 >= call && f2 >= call)) break;
                    if ((unsigned)__builtin_amdgcn_readfirstlane(__hip_atomic_load(ws + WS_TMO, __ATOMIC_RELAXED, __HIP_MEMORY_SCOPE_AGENT)) == call || __builtin_amdgcn_s_memrealtime() - t0 > DEADLINE) { to = true; break; }
                    __builtin_amdgcn_s_sleep(1);
                }
            }
            __builtin_amdgcn_fence(__ATOMIC_ACQUIRE, "");
            if (to && lane == 0) {
                __hip_atomic_store(ws + WS_TMO, call, __ATOMIC_RELAXED, __HIP_MEMORY_SCOPE_AGENT);
                ((unsigned short*)p.out_all[p.rank])[(size_t)p.rank * MLOC * NPR] = 0x7fc0;
            }
        }
        TS(15);
    } else {
        const int li = wid - 4;
        const bool isA = li < 2;
        const int half = li & 1;
        unsigned voff[NLD];
        if (isA) {
#pragma unroll
            for (int j = 0; j < NLD; ++j) { const int row = (half * NLD + j) * 8 + (lane >> 3); voff[j] = (unsigned)(row * KD * 2 + (((lane & 7) ^ ((row >> 1) & 7)) * 16)); }
        } else {
            constexpr int LPR = BROW / 16, RPI = 64 / LPR;
#pragma unroll
            for (int j = 0; j < NLD; ++j) { const int kr = (half * NLD + j) * RPI + lane / LPR; const int c = lane % LPR; const int hh = (kr & 3) | (((kr >> 3) & 1) << 2);
                voff[j] = (unsigned)(kr * ND * 2 + ((c ^ (hh << 1)) * 16)); }
        }
        const unsigned ldso = isA ? (unsigned)(half * NLD * 1024) : (unsigned)(A_BYTES + half * NLD * 1024);
        const size_t kstep = isA ? (size_t)BK * 2 : (size_t)BK * ND * 2;
        int lu = 0, lk = 0; unsigned lslot = 0;
        auto unit_base = [&](int ui) {
            int src, rb, dst, cb; unit_coords(wg, ui < UPW ? ui : 0, p.rank, src, rb, dst, cb);
            if (!isA) return (const char*)p.w + ((size_t)dst * NPR + (size_t)cb * BN) * 2;
            if (src == p.rank || p.single) return (const char*)p.x + (size_t)rb * BM * KD * 2;
            const unsigned call = (unsigned)__builtin_amdgcn_readfirstlane(__hip_atomic_load(ws + WS_TICKET + wg, __ATOMIC_RELAXED, __HIP_MEMORY_SCOPE_AGENT));
            const unsigned long long t0 = __builtin_amdgcn_s_memrealtime();
            const unsigned* fp = ws + WS_XFLAG + 256 * src + lane * 4;
            for (;;) {
                const unsigned f0 = __hip_atomic_load(fp, __ATOMIC_RELAXED, __HIP_MEMORY_SCOPE_SYSTEM), f1 = __hip_atomic_load(fp + 1, __ATOMIC_RELAXED, __HIP_MEMORY_SCOPE_SYSTEM),
                               f2 = __hip_atomic_load(fp + 2, __ATOMIC_RELAXED, __HIP_MEMORY_SCOPE_SYSTEM), f3 = __hip_atomic_load(fp + 3, __ATOMIC_RELAXED, __HIP_MEMORY_SCOPE_SYSTEM);
                if (__all(f0 >= call && f1 >= call && f2 >= call && f3 >= call)) break;
                if ((unsigned)__builtin_amdgcn_readfirstlane(__hip_atomic_load(ws + WS_TMO, __ATOMIC_RELAXED, __HIP_MEMORY_SCOPE_AGENT)) == call
                    || __builtin_amdgcn_s_memrealtime() - t0 > DEADLINE) {
                    __hip_atomic_store(ws + WS_TMO, call, __ATOMIC_RELAXED, __HIP_MEMORY_SCOPE_AGENT);
                    break;
                }
                __builtin_amdgcn_s_sleep(2);
            }
            __builtin_amdgcn_fence(__ATOMIC_ACQUIRE, "");
            return (const char*)ws + WS_XSTAGE + ((size_t)(call & 1) * NR + src) * XBLK_BYTES;
        };
        const char* base = unit_base(0);
#define ISSUE() do { \
        _Pragma("unroll") for (int j = 0; j < NLD; ++j) __builtin_amdgcn_global_load_lds((const unsigned*)(base + voff[j]), (LAS unsigned*)(lds + lslot + ldso + j * 1024), 16, 0, 0); \
        lslot = (lslot + STAGE_BYTES == S * STAGE_BYTES) ? 0u : lslot + STAGE_BYTES; \
        if (++lk == NKT) { lk = 0; ++lu; base = unit_base(lu); } else base += kstep; } while (0)
#define WAITN(n) asm volatile("s_waitcnt vmcnt(%0)" :: "n"((n) * NLD) : "memory")
#pragma unroll
        for (int i = 0; i < S - 1; ++i) ISSUE();
        WAITN(S - 3);
        bar();
#pragma unroll 1
        for (int g = 0; g < G; ++g) {
            if (g + S - 1 < G) { ISSUE(); WAITN(S - 3); }
            else asm volatile("s_waitcnt vmcnt(0)" ::: "memory");
            bar();
        }
    }
}
}

extern "C" void kernel_launch(int rank, int nranks, ncclComm_t comm, hipStream_t stream,
                              void* const* d_in, const int* in_sizes, int n_in,
                              void* d_out, int out_size, const int* in_dtypes, int out_dtype,
                              void* const* d_out_all, void* const* d_ws_all, size_t ws_bytes) {
    static bool init[16] = {false};
    int dev = 0; HIP_CHECK(hipGetDevice(&dev));
    if (!init[dev & 15]) {
        HIP_CHECK(hipFuncSetAttribute((const void*)fused_gemm_a2a, hipFuncAttributeMaxDynamicSharedMemorySize, LDS_BYTES));
        init[dev & 15] = true;
    }
    Params p;
    p.x = (const unsigned short*)d_in[0]; p.w = (const unsigned short*)d_in[1];
    for (int i = 0; i < 8; ++i) { p.out_all[i] = d_out_all[i]; p.ws_all[i] = d_ws_all[i]; }
    p.rank = rank;
    p.single = 0;
    hipLaunchKernelGGL(fused_gemm_a2a, dim3(NWG), dim3(NTHREADS), LDS_BYTES, stream, p);
    HIP_CHECK(hipGetLastError());
}
```

```cpp
#include <hip/hip_runtime.h>
#include <rccl/rccl.h>
#include <cstdio>

#define HIP_CHECK(expr) do { hipError_t _e = (expr); if (_e != hipSuccess) { \
    fprintf(stderr, "HIP error %s at %s:%d\n", hipGetErrorString(_e), __FILE__, __LINE__); \
    return; } } while (0)

namespace {
#define LAS __attribute__((address_space(3)))
typedef short bf16x8 __attribute__((ext_vector_type(8)));
typedef short s16x4 __attribute__((ext_vector_type(4)));
typedef float f32x4 __attribute__((ext_vector_type(4)));
typedef unsigned u32x4 __attribute__((ext_vector_type(4)));

#define NSTAGE_ 5
#define NX0_ 12
#define SEGY_ 6
#define SEGX1_ 5
#define SYNCT_ 4
#define KB0_ 4
#define KB1_ 4
constexpr int MLOC = 512, KD = 4096, ND = 16384, NR = 8, NPR = 2048;
constexpr int BM = 128, BN = 128, BK = 64, S = NSTAGE_;
constexpr int NKT = KD / BK;
constexpr int A_BYTES = BM * BK * 2, B_BYTES = BK * BN * 2, STAGE_BYTES = A_BYTES + B_BYTES;
constexpr int LDS_BYTES = S * STAGE_BYTES;
constexpr int NWG = 256;
constexpr int WM = 64, WN = 64, MI = 4, NI = 4;
constexpr int NLD = 8;
constexpr int NTHREADS = 8 * 64;
constexpr int BROW = BN * 2;
constexpr int G = 2 * NKT;
constexpr int NX0 = NX0_, NX1 = 14 - NX0_;
constexpr int WT = 17;
constexpr unsigned long long DEADLINE = 50000000ull;
static_assert(NX0 >= 9 && NX0 <= 14, "the lower-K half is flagged at slot 8");
static_assert(NX1 * SEGX1_ + 8 * SEGY_ + SYNCT_ < NKT, "slots must fit unit 1");

constexpr int WS_TMO = 64;
constexpr int WS_TICKET = 1024;
constexpr int WS_YFLAG = 4096;
constexpr int WS_XFLAG = 8192;
constexpr size_t WS_XSTAGE = 1u << 20;
constexpr size_t XBLK_BYTES = (size_t)256 * KD * 2;

struct Params {
    const unsigned short* x; const unsigned short* w;
    void* out_all[8]; void* ws_all[8];
    int rank; int single;
};

__device__ __forceinline__ void bar() { asm volatile("" ::: "memory"); __builtin_amdgcn_s_barrier(); asm volatile("" ::: "memory"); }
__device__ __forceinline__ unsigned cvt_pk_bf16(float lo, float hi) { unsigned r; asm("v_cvt_pk_bf16_f32 %0, %1, %2" : "=v"(r) : "v"(lo), "v"(hi)); return r; }
__device__ __forceinline__ float silu(float y) {
    const float c = __builtin_amdgcn_fmed3f(y, -60.f, 60.f);
    const float e = __builtin_amdgcn_exp2f(c * -1.44269504088896340736f);
    return y * __builtin_amdgcn_rcpf(1.0f + e);
}
__device__ __forceinline__ u32x4 pack8(const f32x4 va, const f32x4 vb) {
    const unsigned ax = cvt_pk_bf16(silu(va[0]), silu(va[1])), ay = cvt_pk_bf16(silu(va[2]), silu(va[3]));
    const unsigned bx = cvt_pk_bf16(silu(vb[0]), silu(vb[1])), by = cvt_pk_bf16(silu(vb[2]), silu(vb[3]));
    const auto sx = __builtin_amdgcn_permlane16_swap(ax, bx, false, false);
    const auto sy = __builtin_amdgcn_permlane16_swap(ay, by, false, false);
    return (u32x4){sx[0], sy[0], sx[1], sy[1]};
}

__device__ __forceinline__ void unit_coords(int wg, int ui, int rank, int& src, int& rb, int& dst, int& cb) {
    const int xcd = wg & 7, idx = wg >> 3;
    if (ui == 0) {
        const int u = xcd * 32 + idx, seq = u >> 1;
        src = rank; rb = u & 1;
        if (u < 224) { dst = (rank + 1 + seq % 7) & 7; cb = seq / 7; } else { dst = rank; cb = seq - 112; }
    } else {
        const int r = idx & 3;
        dst = rank; cb = (xcd & 1) * 8 + (idx >> 2);
        src = (rank + (xcd >> 1) * 2 + (r >> 1)) & 7; rb = 2 + (r & 1);
    }
}

#define TS(i) do {} while (0)

__global__ void __launch_bounds__(NTHREADS, 2) fused_gemm_a2a(const Params p) {
    extern __shared__ __attribute__((aligned(16))) unsigned char smem[];
    LAS unsigned char* lds = (LAS unsigned char*)smem;
    const int tid = threadIdx.x, wid = __builtin_amdgcn_readfirstlane(tid >> 6), lane = tid & 63;
    const int wg = blockIdx.x;
    unsigned* ws = (unsigned*)p.ws_all[p.rank];
    TS(0);

    if (wid < 4) {
        const int wr = wid >> 1, wc = wid & 1;
        const int fr = lane & 15, fq = lane >> 4;
        const unsigned ticket = __hip_atomic_load(ws + WS_TICKET + wg, __ATOMIC_RELAXED, __HIP_MEMORY_SCOPE_AGENT);
        const unsigned xo = (unsigned)(wg * 8192 + wid * 1024 + lane * 16);
        const u32x4 xv0 = *(const u32x4*)((const char*)p.x + XBLK_BYTES + xo), xv1 = *(const u32x4*)((const char*)p.x + XBLK_BYTES + xo + 4096);
        const unsigned long long tws = (unsigned long long)p.ws_all[lane & 7], tout = (unsigned long long)p.out_all[lane & 7];
        auto tab = [&](unsigned long long t, int r) {
            const unsigned lo = __builtin_amdgcn_readlane((unsigned)t, r & 7), hi = __builtin_amdgcn_readlane((unsigned)(t >> 32), r & 7);
            return (char*)(((unsigned long long)hi << 32) | lo);
        };
        const unsigned a0 = (unsigned)((wr * WM + fr) * 128 + ((fq ^ (fr >> 1)) * 16)), a1 = a0 ^ 64u;
        const int q = fr >> 2, pp = lane & 3, h = q | ((fq & 1) << 2);
        unsigned bo[NI];
#pragma unroll
        for (int ni = 0; ni < NI; ++ni) bo[ni] = (unsigned)(A_BYTES + (8 * fq + q) * BROW + (((wc * NI + ni) ^ h) * 32) + pp * 8);
        f32x4 ac0[MI][NI], ac1[MI][NI];
#pragma unroll
        for (int mi = 0; mi < MI; ++mi)
#pragma unroll
            for (int ni = 0; ni < NI; ++ni) { ac0[mi][ni] = (f32x4){0.f, 0.f, 0.f, 0.f}; ac1[mi][ni] = (f32x4){0.f, 0.f, 0.f, 0.f}; }
        bf16x8 af[2][MI], bf[2][NI];
#define LOADF(set, sbase, ks) do { \
        _Pragma("unroll") for (int mi = 0; mi < MI; ++mi) af[set][mi] = *(const LAS bf16x8*)(lds + (sbase) + ((ks) ? a1 : a0) + mi * 2048); \
        _Pragma("unroll") for (int ni = 0; ni < NI; ++ni) { \
            s16x4 lo = __builtin_amdgcn_ds_read_tr16_b64_v4i16((LAS s16x4*)(lds + (sbase) + bo[ni] + (ks) * 32 * BROW)); \
            s16x4 hi = __builtin_amdgcn_ds_read_tr16_b64_v4i16((LAS s16x4*)(lds + (sbase) + bo[ni] + (ks) * 32 * BROW + 4 * BROW)); \
            bf[set][ni] = __builtin_shufflevector(lo, hi, 0, 1, 2, 3, 4, 5, 6, 7); } } while (0)
#define MMA(acc, set) do { \
        _Pragma("unroll") for (int mi = 0; mi < MI; ++mi) _Pragma("unroll") for (int ni = 0; ni < NI; ++ni) \
            acc[mi][ni] = __builtin_amdgcn_mfma_f32_16x16x32_bf16(bf[set][ni], af[set][mi], acc[mi][ni], 0, 0, 0); } while (0)
#define INTERLEAVE() do { \
        _Pragma("unroll") for (int i = 0; i < MI + 2 * NI; ++i) { __builtin_amdgcn_sched_group_barrier(0x8, 1, 0); __builtin_amdgcn_sched_group_barrier(0x100, 1, 0); } \
        __builtin_amdgcn_sched_group_barrier(0x8, MI * NI - (MI + 2 * NI), 0); } while (0)
#define TILE(acc) do { \
            const unsigned sn = (sb + STAGE_BYTES == S * STAGE_BYTES) ? 0u : sb + STAGE_BYTES; \
            LOADF(1, sb, 1); MMA(acc, 0); INTERLEAVE(); \
            LOADF(0, sn, 0); MMA(acc, 1); INTERLEAVE(); \
            sb = sn; } while (0)
#define INTERLEAVE_V() do { \
        _Pragma("unroll") for (int i = 0; i < MI + 2 * NI; ++i) { __builtin_amdgcn_sched_group_barrier(0x8, 1, 0); __builtin_amdgcn_sched_group_barrier(0x100, 1, 0); __builtin_amdgcn_sched_group_barrier(0x402, 2, 0); } \
        _Pragma("unroll") for (int i = 0; i < MI * NI - (MI + 2 * NI); ++i) { __builtin_amdgcn_sched_group_barrier(0x8, 1, 0); __builtin_amdgcn_sched_group_barrier(0x402, 3, 0); } } while (0)
        bar();
        TS(1);
        LOADF(0, 0u, 0);
        unsigned sb = 0;
        unsigned call = 0;
        auto push_x = [&](int sl) {
            const int pr = (p.rank + 1 + (sl + wg + wid) % 7) & 7;
            const __amdgpu_buffer_rsrc_t xr = __builtin_amdgcn_make_buffer_rsrc(tab(tws, pr) + WS_XSTAGE + ((size_t)(call & 1) * NR + p.rank) * XBLK_BYTES, 0, (int)XBLK_BYTES, 0x00020000);
            if (sl < 7) __builtin_amdgcn_raw_buffer_store_b128(xv0, xr, xo, 0, WT);
            else __builtin_amdgcn_raw_buffer_store_b128(xv1, xr, xo + 4096, 0, WT);
        };
        auto flag_x = [&](int kh) {
            if (wid == 0 && lane < 8 && lane != p.rank && !p.single)
                __hip_atomic_store((unsigned*)tws + WS_XFLAG + 512 * p.rank + 256 * kh + wg, call, __ATOMIC_RELAXED, __HIP_MEMORY_SCOPE_SYSTEM);
        };
        {
            int done = 0;
#pragma unroll 1
            for (int sl = 0; sl < NX0; ++sl) {
                const int nt = ((sl + 1) * NKT) / NX0 - done;
                done += nt;
#pragma unroll 1
                for (int t = 0; t < nt; ++t) {
                    TILE(ac0);
                    if (t == nt - 1) {
                        if (sl == 0) call = (unsigned)__builtin_amdgcn_readfirstlane(ticket) + 1u;
                        if (sl == 1 && tid == 0) __hip_atomic_store(ws + WS_TICKET + wg, call, __ATOMIC_RELAXED, __HIP_MEMORY_SCOPE_AGENT);
                        if (sl == 8) asm volatile("s_waitcnt vmcnt(1)" ::: "memory");
                        if (!p.single) push_x(sl);
                    }
                    bar();
                }
                if (sl == 8) flag_x(0);
                if (sl == 9) TS(9);
            }
        }
        TS(2); TS(3);
        int src, rb, dst, cb; unit_coords(wg, 0, p.rank, src, rb, dst, cb);
        dst = __builtin_amdgcn_readfirstlane(dst) & 7;
        if (p.single) dst = p.rank;
        const unsigned lofs = (unsigned)(((wr * WM + fr) * NPR + wc * WN + (fq & 1) * 16 + (fq >> 1) * 8) * 2);
        unsigned tmo;
        {
            const __amdgpu_buffer_rsrc_t rs = __builtin_amdgcn_make_buffer_rsrc(tab(tout, dst), 0, 4096 * NPR * 2, 0x00020000);
            const unsigned gofs = (unsigned)(((p.rank * MLOC + rb * BM) * NPR + cb * BN) * 2) + lofs;
#pragma unroll 1
            for (int sl = NX0; sl < 14; ++sl) {
#pragma unroll 1
                for (int t = 0; t < SEGX1_; ++t) {
                    TILE(ac1);
                    if (t == SEGX1_ - 1 && !p.single) push_x(sl);
                    bar();
                }
            }
#pragma unroll
            for (int seg = 0; seg < 8; ++seg) {
#pragma unroll 1
                for (int t = 0; t < SEGY_ - 1; ++t) { TILE(ac1); bar(); }
                {
                    const unsigned sn = (sb + STAGE_BYTES == S * STAGE_BYTES) ? 0u : sb + STAGE_BYTES;
                    const u32x4 o = pack8(ac0[seg >> 1][2 * (seg & 1)], ac0[seg >> 1][2 * (seg & 1) + 1]);
                    LOADF(1, sb, 1); MMA(ac1, 0); INTERLEAVE_V();
                    LOADF(0, sn, 0); MMA(ac1, 1); INTERLEAVE_V();
                    sb = sn;
                    if (seg == 0) asm volatile("s_waitcnt vmcnt(0)" ::: "memory");
                    __builtin_amdgcn_raw_buffer_store_b128(o, rs, gofs + (unsigned)(((seg >> 1) * 16 * NPR + (seg & 1) * 32) * 2), 0, WT);
                    bar();
                }
                if (seg == 0) flag_x(1);
            }
#pragma unroll 1
            for (int t = 0; t < SYNCT_ - 1; ++t) { TILE(ac1); bar(); }
            TILE(ac1);
            asm volatile("s_waitcnt vmcnt(0)" ::: "memory");
            if (dst != p.rank && lane == dst)
                __hip_atomic_store((unsigned*)tws + WS_YFLAG + 256 * p.rank + (cb * 2 + rb) * 4 + wid, call, __ATOMIC_RELAXED, __HIP_MEMORY_SCOPE_SYSTEM);
            tmo = __hip_atomic_load(ws + WS_TMO, __ATOMIC_RELAXED, __HIP_MEMORY_SCOPE_AGENT);
            bar();
#pragma unroll 1
            for (int t = 0; t < NKT - NX1 * SEGX1_ - 8 * SEGY_ - SYNCT_; ++t) { TILE(ac1); bar(); }
        }
        TS(4);
        {
            unit_coords(wg, 1, p.rank, src, rb, dst, cb);
            src = __builtin_amdgcn_readfirstlane(src) & 7;
            const bool bad = src != p.rank && (unsigned)__builtin_amdgcn_readfirstlane(tmo) == call;
            const __amdgpu_buffer_rsrc_t rs = __builtin_amdgcn_make_buffer_rsrc(p.out_all[p.rank], 0, 4096 * NPR * 2, 0x00020000);
            const unsigned gofs = (unsigned)(((src * MLOC + rb * BM) * NPR + cb * BN) * 2) + lofs;
#pragma unroll
            for (int i = 0; i < 8; ++i) {
                u32x4 o = pack8(ac1[i >> 1][2 * (i & 1)], ac1[i >> 1][2 * (i & 1) + 1]);
                if (bad) o = (u32x4){0x7fc07fc0u, 0x7fc07fc0u, 0x7fc07fc0u, 0x7fc07fc0u};
                __builtin_amdgcn_raw_buffer_store_b128(o, rs, gofs + (unsigned)(((i >> 1) * 16 * NPR + (i & 1) * 32) * 2), 0, WT);
            }
        }
        TS(14);
        if (wg < 7 * 8 && (wg & 7) == 0 && wid == 0 && !p.single) {
            const unsigned* fp = ws + WS_YFLAG + 256 * ((p.rank + 1 + (wg >> 3)) & 7) + lane * 2;
            const unsigned long long t0 = __builtin_amdgcn_s_memrealtime();
            bool to = false;
            for (;;) {
                const unsigned f0 = __hip_atomic_load(fp, __ATOMIC_RELAXED, __HIP_MEMORY_SCOPE_SYSTEM), f1 = __hip_atomic_load(fp + 1, __ATOMIC_RELAXED, __HIP_MEMORY_SCOPE_SYSTEM);
                if (__all(f0 >= call && f1 >= call)) break;
                if ((unsigned)__builtin_amdgcn_readfirstlane(__hip_atomic_load(ws + WS_TMO, __ATOMIC_RELAXED, __HIP_MEMORY_SCOPE_AGENT)) == call
                    || __builtin_amdgcn_s_memrealtime() - t0 > DEADLINE) { to = true; break; }
                __builtin_amdgcn_s_sleep(1);
            }
            __builtin_amdgcn_fence(__ATOMIC_ACQUIRE, "");
            if (to) {
                asm volatile("s_waitcnt vmcnt(0)" ::: "memory");
                if (lane == 0) {
                    __hip_atomic_store(ws + WS_TMO, call, __ATOMIC_RELAXED, __HIP_MEMORY_SCOPE_AGENT);
                    ((unsigned short*)p.out_all[p.rank])[((size_t)(src * MLOC + rb * BM)) * NPR + cb * BN] = 0x7fc0;
                }
            }
        }
        TS(15);
    } else {
        const int li = wid - 4;
        const bool isA = li < 2;
        const int half = li & 1;
        unsigned voff[NLD];
        if (isA) {
#pragma unroll
            for (int j = 0; j < NLD; ++j) { const int row = (half * NLD + j) * 8 + (lane >> 3); voff[j] = (unsigned)(row * KD * 2 + (((lane & 7) ^ ((row >> 1) & 7)) * 16)); }
        } else {
#pragma unroll
            for (int j = 0; j < NLD; ++j) { const int kr = (half * NLD + j) * 4 + (lane >> 4); const int hh = (kr & 3) | (((kr >> 3) & 1) << 2);
                voff[j] = (unsigned)(kr * ND * 2 + (((lane & 15) ^ (hh << 1)) * 16)); }
        }
        const unsigned ldso = isA ? (unsigned)(half * NLD * 1024) : (unsigned)(A_BYTES + half * NLD * 1024);
        const size_t kstep = isA ? (size_t)BK * 2 : (size_t)BK * ND * 2;
        int lu = 0, lk = 0; unsigned lslot = 0;
        int gsrc = -1; unsigned call = 0;
        auto wait_x = [&](int kh) {
            const unsigned long long t0 = __builtin_amdgcn_s_memrealtime();
            const unsigned* fp = ws + WS_XFLAG + 512 * gsrc + 256 * kh + lane * 4;
            for (;;) {
                const unsigned f0 = __hip_atomic_load(fp, __ATOMIC_RELAXED, __HIP_MEMORY_SCOPE_SYSTEM), f1 = __hip_atomic_load(fp + 1, __ATOMIC_RELAXED, __HIP_MEMORY_SCOPE_SYSTEM),
                               f2 = __hip_atomic_load(fp + 2, __ATOMIC_RELAXED, __HIP_MEMORY_SCOPE_SYSTEM), f3 = __hip_atomic_load(fp + 3, __ATOMIC_RELAXED, __HIP_MEMORY_SCOPE_SYSTEM);
                if (__all(f0 >= call && f1 >= call && f2 >= call && f3 >= call)) break;
                if ((unsigned)__builtin_amdgcn_readfirstlane(__hip_atomic_load(ws + WS_TMO, __ATOMIC_RELAXED, __HIP_MEMORY_SCOPE_AGENT)) == call
                    || __builtin_amdgcn_s_memrealtime() - t0 > DEADLINE) {
                    __hip_atomic_store(ws + WS_TMO, call, __ATOMIC_RELAXED, __HIP_MEMORY_SCOPE_AGENT);
                    break;
                }
                __builtin_amdgcn_s_sleep(2);
            }
            __builtin_amdgcn_fence(__ATOMIC_ACQUIRE, "wavefront");
        };
        auto unit_base = [&](int ui) {
            int src, rb, dst, cb; unit_coords(wg, ui, p.rank, src, rb, dst, cb);
            gsrc = -1;
            if (!isA) return (const char*)p.w + ((size_t)dst * NPR + (size_t)cb * BN) * 2;
            if (src == p.rank || p.single) return (const char*)p.x + (size_t)rb * BM * KD * 2;
            call = (unsigned)__builtin_amdgcn_readfirstlane(__hip_atomic_load(ws + WS_TICKET + wg, __ATOMIC_RELAXED, __HIP_MEMORY_SCOPE_AGENT));
            gsrc = src; wait_x(0);
            return (const char*)ws + WS_XSTAGE + ((size_t)(call & 1) * NR + src) * XBLK_BYTES + (size_t)(rb - 2) * BM * KD * 2;
        };
        const int idx = wg >> 3;
        int rot = (idx & 1) * (KB0_ / 2) + (idx >> 1), kbm = KB0_ - 1;
        const char* ubase = unit_base(0);
#define ISSUE() do { \
        const char* base = ubase + (size_t)((lk & ~kbm) + ((lk + rot) & kbm)) * kstep; \
        _Pragma("unroll") for (int j = 0; j < NLD; ++j) __builtin_amdgcn_global_load_lds((const unsigned*)(base + voff[j]), (LAS unsigned*)(lds + lslot + ldso + j * 1024), 16, 0, 0); \
        lslot = (lslot + STAGE_BYTES == S * STAGE_BYTES) ? 0u : lslot + STAGE_BYTES; \
        if (++lk == NKT) { lk = 0; if (++lu == 1) { ubase = unit_base(1); rot = (idx & 3) * (KB1_ / 4) + (idx >> 2); kbm = KB1_ - 1; } } \
        else if (lk == NKT / 2 && gsrc >= 0) wait_x(1); } while (0)
#define WAITN(n) asm volatile("s_waitcnt vmcnt(%0)" :: "n"((n) * NLD) : "memory")
#pragma unroll
        for (int i = 0; i < S - 1; ++i) ISSUE();
        WAITN(S - 3);
        bar();
#pragma unroll 1
        for (int g = 0; g < G; ++g) {
            if (g + S - 1 < G) { ISSUE(); WAITN(S - 3); }
            else asm volatile("s_waitcnt vmcnt(0)" ::: "memory");
            bar();
        }
    }
}
}

extern "C" void kernel_launch(int rank, int nranks, ncclComm_t comm, hipStream_t stream,
                              void* const* d_in, const int* in_sizes, int n_in,
                              void* d_out, int out_size, const int* in_dtypes, int out_dtype,
                              void* const* d_out_all, void* const* d_ws_all, size_t ws_bytes) {
    if (nranks != NR || (unsigned)rank >= (unsigned)NR || ws_bytes < WS_XSTAGE + 2 * NR * XBLK_BYTES) { fprintf(stderr, "unexpected configuration\n"); return; }
    static bool init[16] = {false};
    int dev = 0; HIP_CHECK(hipGetDevice(&dev));
    if (!init[dev & 15]) {
        HIP_CHECK(hipFuncSetAttribute((const void*)fused_gemm_a2a, hipFuncAttributeMaxDynamicSharedMemorySize, LDS_BYTES));
        init[dev & 15] = true;
    }
    Params p;
    p.x = (const unsigned short*)d_in[0]; p.w = (const unsigned short*)d_in[1];
    for (int i = 0; i < 8; ++i) { p.out_all[i] = d_out_all[i]; p.ws_all[i] = d_ws_all[i]; }
    p.rank = rank;
    p.single = 0;
    hipLaunchKernelGGL(fused_gemm_a2a, dim3(NWG), dim3(NTHREADS), LDS_BYTES, stream, p);
    HIP_CHECK(hipGetLastError());
}
```

```cpp
#include <hip/hip_runtime.h>
#include <rccl/rccl.h>
#include <cstdio>

#define HIP_CHECK(expr) do { hipError_t _e = (expr); if (_e != hipSuccess) { \
    fprintf(stderr, "HIP error %s at %s:%d\n", hipGetErrorString(_e), __FILE__, __LINE__); \
    return; } } while (0)

namespace {
#define LAS __attribute__((address_space(3)))
typedef short bf16x8 __attribute__((ext_vector_type(8)));
typedef short s16x4 __attribute__((ext_vector_type(4)));
typedef float f32x4 __attribute__((ext_vector_type(4)));
typedef unsigned u32x4 __attribute__((ext_vector_type(4)));

#define NSTAGE_ 5
#define NX0_ 12
#define XB_ 1
#define SEGY_ 6
#define SEGX1_ 5
#define SYNCT_ 4
#define KB0_ 4
#define KB1_ 4
constexpr int MLOC = 512, KD = 4096, ND = 16384, NR = 8, NPR = 2048;
constexpr int BM = 128, BN = 128, BK = 64, S = NSTAGE_;
constexpr int NKT = KD / BK;
constexpr int A_BYTES = BM * BK * 2, B_BYTES = BK * BN * 2, STAGE_BYTES = A_BYTES + B_BYTES;
constexpr int LDS_BYTES = S * STAGE_BYTES;
constexpr int NWG = 256;
constexpr int WM = 64, WN = 64, MI = 4, NI = 4;
constexpr int NLD = 8;
constexpr int NTHREADS = 8 * 64;
constexpr int BROW = BN * 2;
constexpr int G = 2 * NKT;
constexpr int NX0 = NX0_, NX1 = 14 - NX0_;
constexpr int WT = 17;
constexpr unsigned long long DEADLINE = 50000000ull;
static_assert(NX0 >= 9 && NX0 <= 14, "the lower-K half is flagged at slot 8");
static_assert(NX1 * SEGX1_ + 8 * SEGY_ + SYNCT_ < NKT, "slots must fit unit 1");

constexpr int WS_TMO = 64;
constexpr int WS_TICKET = 1024;
constexpr int WS_YFLAG = 4096;
constexpr int WS_XFLAG = 8192;
constexpr size_t WS_XSTAGE = 1u << 20;
constexpr size_t XBLK_BYTES = (size_t)256 * KD * 2;

struct Params {
    const unsigned short* x; const unsigned short* w;
    void* out_all[8]; void* ws_all[8];
    int rank; int single;
};

__device__ __forceinline__ void bar() { asm volatile("" ::: "memory"); __builtin_amdgcn_s_barrier(); asm volatile("" ::: "memory"); }
__device__ __forceinline__ unsigned cvt_pk_bf16(float lo, float hi) { unsigned r; asm("v_cvt_pk_bf16_f32 %0, %1, %2" : "=v"(r) : "v"(lo), "v"(hi)); return r; }
__device__ __forceinline__ float silu(float y) {
    const float c = __builtin_amdgcn_fmed3f(y, -60.f, 60.f);
    const float e = __builtin_amdgcn_exp2f(c * -1.44269504088896340736f);
    return y * __builtin_amdgcn_rcpf(1.0f + e);
}
__device__ __forceinline__ u32x4 pack8(const f32x4 va, const f32x4 vb) {
    const unsigned ax = cvt_pk_bf16(silu(va[0]), silu(va[1])), ay = cvt_pk_bf16(silu(va[2]), silu(va[3]));
    const unsigned bx = cvt_pk_bf16(silu(vb[0]), silu(vb[1])), by = cvt_pk_bf16(silu(vb[2]), silu(vb[3]));
    const auto sx = __builtin_amdgcn_permlane16_swap(ax, bx, false, false);
    const auto sy = __builtin_amdgcn_permlane16_swap(ay, by, false, false);
    return (u32x4){sx[0], sy[0], sx[1], sy[1]};
}

__device__ __forceinline__ void unit_coords(int wg, int ui, int rank, int& src, int& rb, int& dst, int& cb) {
    const int xcd = wg & 7, idx = wg >> 3;
    if (ui == 0) {
        const int u = xcd * 32 + idx, seq = u >> 1;
        src = rank; rb = u & 1;
        if (u < 224) { dst = (rank + 1 + seq % 7) & 7; cb = seq / 7; } else { dst = rank; cb = seq - 112; }
    } else {
        const int r = idx & 3;
        dst = rank; cb = (xcd & 1) * 8 + (idx >> 2);
        src = (rank + (xcd >> 1) * 2 + (r >> 1)) & 7; rb = 2 + (r & 1);
    }
}

#define TS(i) do {} while (0)

__global__ void __launch_bounds__(NTHREADS, 2) fused_gemm_a2a(const Params p) {
    extern __shared__ __attribute__((aligned(16))) unsigned char smem[];
    LAS unsigned char* lds = (LAS unsigned char*)smem;
    const int tid = threadIdx.x, wid = __builtin_amdgcn_readfirstlane(tid >> 6), lane = tid & 63;
    const int wg = blockIdx.x;
    unsigned* ws = (unsigned*)p.ws_all[p.rank];
    TS(0);

    if (wid < 4) {
        const int wr = wid >> 1, wc = wid & 1;
        const int fr = lane & 15, fq = lane >> 4;
        const unsigned ticket = __hip_atomic_load(ws + WS_TICKET + wg, __ATOMIC_RELAXED, __HIP_MEMORY_SCOPE_AGENT);
        const unsigned xo = (unsigned)(wg * 8192 + wid * 1024 + lane * 16);
        const u32x4 xv0 = *(const u32x4*)((const char*)p.x + XBLK_BYTES + xo), xv1 = *(const u32x4*)((const char*)p.x + XBLK_BYTES + xo + 4096);
        const unsigned long long tws = (unsigned long long)p.ws_all[lane & 7], tout = (unsigned long long)p.out_all[lane & 7];
        auto tab = [&](unsigned long long t, int r) {
            const unsigned lo = __builtin_amdgcn_readlane((unsigned)t, r & 7), hi = __builtin_amdgcn_readlane((unsigned)(t >> 32), r & 7);
            return (char*)(((unsigned long long)hi << 32) | lo);
        };
        const unsigned a0 = (unsigned)((wr * WM + fr) * 128 + ((fq ^ (fr >> 1)) * 16)), a1 = a0 ^ 64u;
        const int q = fr >> 2, pp = lane & 3, h = q | ((fq & 1) << 2);
        unsigned bo[NI];
#pragma unroll
        for (int ni = 0; ni < NI; ++ni) bo[ni] = (unsigned)(A_BYTES + (8 * fq + q) * BROW + (((wc * NI + ni) ^ h) * 32) + pp * 8);
        f32x4 ac0[MI][NI], ac1[MI][NI];
#pragma unroll
        for (int mi = 0; mi < MI; ++mi)
#pragma unroll
            for (int ni = 0; ni < NI; ++ni) { ac0[mi][ni] = (f32x4){0.f, 0.f, 0.f, 0.f}; ac1[mi][ni] = (f32x4){0.f, 0.f, 0.f, 0.f}; }
        bf16x8 af[2][MI], bf[2][NI];
#define LOADF(set, sbase, ks) do { \
        _Pragma("unroll") for (int mi = 0; mi < MI; ++mi) af[set][mi] = *(const LAS bf16x8*)(lds + (sbase) + ((ks) ? a1 : a0) + mi * 2048); \
        _Pragma("unroll") for (int ni = 0; ni < NI; ++ni) { \
            s16x4 lo = __builtin_amdgcn_ds_read_tr16_b64_v4i16((LAS s16x4*)(lds + (sbase) + bo[ni] + (ks) * 32 * BROW)); \
            s16x4 hi = __builtin_amdgcn_ds_read_tr16_b64_v4i16((LAS s16x4*)(lds + (sbase) + bo[ni] + (ks) * 32 * BROW + 4 * BROW)); \
            bf[set][ni] = __builtin_shufflevector(lo, hi, 0, 1, 2, 3, 4, 5, 6, 7); } } while (0)
#define MMA(acc, set) do { \
        _Pragma("unroll") for (int mi = 0; mi < MI; ++mi) _Pragma("unroll") for (int ni = 0; ni < NI; ++ni) \
            acc[mi][ni] = __builtin_amdgcn_mfma_f32_16x16x32_bf16(bf[set][ni], af[set][mi], acc[mi][ni], 0, 0, 0); } while (0)
#define INTERLEAVE() do { \
        _Pragma("unroll") for (int i = 0; i < MI + 2 * NI; ++i) { __builtin_amdgcn_sched_group_barrier(0x8, 1, 0); __builtin_amdgcn_sched_group_barrier(0x100, 1, 0); } \
        __builtin_amdgcn_sched_group_barrier(0x8, MI * NI - (MI + 2 * NI), 0); } while (0)
#define TILE(acc) do { \
            const unsigned sn = (sb + STAGE_BYTES == S * STAGE_BYTES) ? 0u : sb + STAGE_BYTES; \
            LOADF(1, sb, 1); MMA(acc, 0); INTERLEAVE(); \
            LOADF(0, sn, 0); MMA(acc, 1); INTERLEAVE(); \
            sb = sn; } while (0)
#define INTERLEAVE_V() do { \
        _Pragma("unroll") for (int i = 0; i < MI + 2 * NI; ++i) { __builtin_amdgcn_sched_group_barrier(0x8, 1, 0); __builtin_amdgcn_sched_group_barrier(0x100, 1, 0); __builtin_amdgcn_sched_group_barrier(0x402, 2, 0); } \
        _Pragma("unroll") for (int i = 0; i < MI * NI - (MI + 2 * NI); ++i) { __builtin_amdgcn_sched_group_barrier(0x8, 1, 0); __builtin_amdgcn_sched_group_barrier(0x402, 3, 0); } } while (0)
        bar();
        TS(1);
        LOADF(0, 0u, 0);
        unsigned sb = 0;
        unsigned call = 0;
        auto push_x = [&](int sl) {
            const int pr = (p.rank + 1 + (sl + wg + wid) % 7) & 7;
            const __amdgpu_buffer_rsrc_t xr = __builtin_amdgcn_make_buffer_rsrc(tab(tws, pr) + WS_XSTAGE + ((size_t)(call & 1) * NR + p.rank) * XBLK_BYTES, 0, (int)XBLK_BYTES, 0x00020000);
            if (sl < 7) __builtin_amdgcn_raw_buffer_store_b128(xv0, xr, xo, 0, WT);
            else __builtin_amdgcn_raw_buffer_store_b128(xv1, xr, xo + 4096, 0, WT);
        };
        auto flag_x = [&](int kh) {
            if (wid == 0 && lane < 8 && lane != p.rank && !p.single)
                __hip_atomic_store((unsigned*)tws + WS_XFLAG + 512 * p.rank + 256 * kh + wg, call, __ATOMIC_RELAXED, __HIP_MEMORY_SCOPE_SYSTEM);
        };
        {
            constexpr int NSL = NX0 / XB_;
            constexpr int FS = (7 + XB_ - 1) / XB_ + 1;
            int done = 0;
#pragma unroll 1
            for (int sl = 0; sl < NSL; ++sl) {
                const int nt = ((sl + 1) * NKT) / NSL - done;
                done += nt;
#pragma unroll 1
                for (int t = 0; t < nt; ++t) {
                    TILE(ac0);
                    if (t == nt - 1) {
                        if (sl == 0) call = (unsigned)__builtin_amdgcn_readfirstlane(ticket) + 1u;
                        if (sl == 1 && tid == 0) __hip_atomic_store(ws + WS_TICKET + wg, call, __ATOMIC_RELAXED, __HIP_MEMORY_SCOPE_AGENT);
                        if (sl == FS) asm volatile("s_waitcnt vmcnt(%0)" :: "n"(XB_) : "memory");
                        if (!p.single) {
#pragma unroll
                            for (int b = 0; b < XB_; ++b) push_x(sl * XB_ + b);
                        }
                    }
                    bar();
                }
                if (sl == FS) flag_x(0);
                if (sl == NSL - 2) TS(9);
            }
        }
        TS(2); TS(3);
        int src, rb, dst, cb; unit_coords(wg, 0, p.rank, src, rb, dst, cb);
        dst = __builtin_amdgcn_readfirstlane(dst) & 7;
        if (p.single) dst = p.rank;
        const unsigned lofs = (unsigned)(((wr * WM + fr) * NPR + wc * WN + (fq & 1) * 16 + (fq >> 1) * 8) * 2);
        unsigned tmo;
        {
            const __amdgpu_buffer_rsrc_t rs = __builtin_amdgcn_make_buffer_rsrc(tab(tout, dst), 0, 4096 * NPR * 2, 0x00020000);
            const unsigned gofs = (unsigned)(((p.rank * MLOC + rb * BM) * NPR + cb * BN) * 2) + lofs;
#pragma unroll 1
            for (int sl = NX0; sl < 14; ++sl) {
#pragma unroll 1
                for (int t = 0; t < SEGX1_; ++t) {
                    TILE(ac1);
                    if (t == SEGX1_ - 1 && !p.single) push_x(sl);
                    bar();
                }
            }
#pragma unroll
            for (int seg = 0; seg < 8; ++seg) {
#pragma unroll 1
                for (int t = 0; t < SEGY_ - 1; ++t) { TILE(ac1); bar(); }
                {
                    const unsigned sn = (sb + STAGE_BYTES == S * STAGE_BYTES) ? 0u : sb + STAGE_BYTES;
                    const u32x4 o = pack8(ac0[seg >> 1][2 * (seg & 1)], ac0[seg >> 1][2 * (seg & 1) + 1]);
                    LOADF(1, sb, 1); MMA(ac1, 0); INTERLEAVE_V();
                    LOADF(0, sn, 0); MMA(ac1, 1); INTERLEAVE_V();
                    sb = sn;
                    if (seg == 0) asm volatile("s_waitcnt vmcnt(0)" ::: "memory");
                    __builtin_amdgcn_raw_buffer_store_b128(o, rs, gofs + (unsigned)(((seg >> 1) * 16 * NPR + (seg & 1) * 32) * 2), 0, WT);
                    bar();
                }
                if (seg == 0) flag_x(1);
            }
#pragma unroll 1
            for (int t = 0; t < SYNCT_ - 1; ++t) { TILE(ac1); bar(); }
            TILE(ac1);
            asm volatile("s_waitcnt vmcnt(0)" ::: "memory");
            if (dst != p.rank && lane == dst)
                __hip_atomic_store((unsigned*)tws + WS_YFLAG + 256 * p.rank + (cb * 2 + rb) * 4 + wid, call, __ATOMIC_RELAXED, __HIP_MEMORY_SCOPE_SYSTEM);
            tmo = __hip_atomic_load(ws + WS_TMO, __ATOMIC_RELAXED, __HIP_MEMORY_SCOPE_AGENT);
            bar();
#pragma unroll 1
            for (int t = 0; t < NKT - NX1 * SEGX1_ - 8 * SEGY_ - SYNCT_; ++t) { TILE(ac1); bar(); }
        }
        TS(4);
        {
            unit_coords(wg, 1, p.rank, src, rb, dst, cb);
            src = __builtin_amdgcn_readfirstlane(src) & 7;
            const bool bad = src != p.rank && (unsigned)__builtin_amdgcn_readfirstlane(tmo) == call;
            const __amdgpu_buffer_rsrc_t rs = __builtin_amdgcn_make_buffer_rsrc(p.out_all[p.rank], 0, 4096 * NPR * 2, 0x00020000);
            const unsigned gofs = (unsigned)(((src * MLOC + rb * BM) * NPR + cb * BN) * 2) + lofs;
#pragma unroll
            for (int i = 0; i < 8; ++i) {
                u32x4 o = pack8(ac1[i >> 1][2 * (i & 1)], ac1[i >> 1][2 * (i & 1) + 1]);
                if (bad) o = (u32x4){0x7fc07fc0u, 0x7fc07fc0u, 0x7fc07fc0u, 0x7fc07fc0u};
                __builtin_amdgcn_raw_buffer_store_b128(o, rs, gofs + (unsigned)(((i >> 1) * 16 * NPR + (i & 1) * 32) * 2), 0, WT);
            }
        }
        TS(14);
        if (wg < 7 * 8 && (wg & 7) == 0 && wid == 0 && !p.single) {
            const unsigned* fp = ws + WS_YFLAG + 256 * ((p.rank + 1 + (wg >> 3)) & 7) + lane * 2;
            const unsigned long long t0 = __builtin_amdgcn_s_memrealtime();
            bool to = false;
            for (;;) {
                const unsigned f0 = __hip_atomic_load(fp, __ATOMIC_RELAXED, __HIP_MEMORY_SCOPE_SYSTEM), f1 = __hip_atomic_load(fp + 1, __ATOMIC_RELAXED, __HIP_MEMORY_SCOPE_SYSTEM);
                if (__all(f0 >= call && f1 >= call)) break;
                if ((unsigned)__builtin_amdgcn_readfirstlane(__hip_atomic_load(ws + WS_TMO, __ATOMIC_RELAXED, __HIP_MEMORY_SCOPE_AGENT)) == call
                    || __builtin_amdgcn_s_memrealtime() - t0 > DEADLINE) { to = true; break; }
                __builtin_amdgcn_s_sleep(1);
            }
            __builtin_amdgcn_fence(__ATOMIC_ACQUIRE, "");
            if (to) {
                asm volatile("s_waitcnt vmcnt(0)" ::: "memory");
                if (lane == 0) {
                    __hip_atomic_store(ws + WS_TMO, call, __ATOMIC_RELAXED, __HIP_MEMORY_SCOPE_AGENT);
                    ((unsigned short*)p.out_all[p.rank])[((size_t)(src * MLOC + rb * BM)) * NPR + cb * BN] = 0x7fc0;
                }
            }
        }
        TS(15);
    } else {
        const int li = wid - 4;
        const bool isA = li < 2;
        const int half = li & 1;
        unsigned voff[NLD];
        if (isA) {
#pragma unroll
            for (int j = 0; j < NLD; ++j) { const int row = (half * NLD + j) * 8 + (lane >> 3); voff[j] = (unsigned)(row * KD * 2 + (((lane & 7) ^ ((row >> 1) & 7)) * 16)); }
        } else {
#pragma unroll
            for (int j = 0; j < NLD; ++j) { const int kr = (half * NLD + j) * 4 + (lane >> 4); const int hh = (kr & 3) | (((kr >> 3) & 1) << 2);
                voff[j] = (unsigned)(kr * ND * 2 + (((lane & 15) ^ (hh << 1)) * 16)); }
        }
        const unsigned ldso = isA ? (unsigned)(half * NLD * 1024) : (unsigned)(A_BYTES + half * NLD * 1024);
        const size_t kstep = isA ? (size_t)BK * 2 : (size_t)BK * ND * 2;
        int lu = 0, lk = 0; unsigned lslot = 0;
        int gsrc = -1; unsigned call = 0;
        auto wait_x = [&](int kh) {
            const unsigned long long t0 = __builtin_amdgcn_s_memrealtime();
            const unsigned* fp = ws + WS_XFLAG + 512 * gsrc + 256 * kh + lane * 4;
            for (;;) {
                const unsigned f0 = __hip_atomic_load(fp, __ATOMIC_RELAXED, __HIP_MEMORY_SCOPE_SYSTEM), f1 = __hip_atomic_load(fp + 1, __ATOMIC_RELAXED, __HIP_MEMORY_SCOPE_SYSTEM),
                               f2 = __hip_atomic_load(fp + 2, __ATOMIC_RELAXED, __HIP_MEMORY_SCOPE_SYSTEM), f3 = __hip_atomic_load(fp + 3, __ATOMIC_RELAXED, __HIP_MEMORY_SCOPE_SYSTEM);
                if (__all(f0 >= call && f1 >= call && f2 >= call && f3 >= call)) break;
                if ((unsigned)__builtin_amdgcn_readfirstlane(__hip_atomic_load(ws + WS_TMO, __ATOMIC_RELAXED, __HIP_MEMORY_SCOPE_AGENT)) == call
                    || __builtin_amdgcn_s_memrealtime() - t0 > DEADLINE) {
                    __hip_atomic_store(ws + WS_TMO, call, __ATOMIC_RELAXED, __HIP_MEMORY_SCOPE_AGENT);
                    break;
                }
                __builtin_amdgcn_s_sleep(2);
            }
            __builtin_amdgcn_fence(__ATOMIC_ACQUIRE, "wavefront");
        };
        auto unit_base = [&](int ui) {
            int src, rb, dst, cb; unit_coords(wg, ui, p.rank, src, rb, dst, cb);
            gsrc = -1;
            if (!isA) return (const char*)p.w + ((size_t)dst * NPR + (size_t)cb * BN) * 2;
            if (src == p.rank || p.single) return (const char*)p.x + (size_t)rb * BM * KD * 2;
            call = (unsigned)__builtin_amdgcn_readfirstlane(__hip_atomic_load(ws + WS_TICKET + wg, __ATOMIC_RELAXED, __HIP_MEMORY_SCOPE_AGENT));
            gsrc = src; wait_x(0);
            return (const char*)ws + WS_XSTAGE + ((size_t)(call & 1) * NR + src) * XBLK_BYTES + (size_t)(rb - 2) * BM * KD * 2;
        };
        const int idx = wg >> 3;
        int rot = (idx & 1) * (KB0_ / 2) + (idx >> 1), kbm = KB0_ - 1;
        const char* ubase = unit_base(0);
#define ISSUE() do { \
        const char* base = ubase + (size_t)((lk & ~kbm) + ((lk + rot) & kbm)) * kstep; \
        _Pragma("unroll") for (int j = 0; j < NLD; ++j) __builtin_amdgcn_global_load_lds((const unsigned*)(base + voff[j]), (LAS unsigned*)(lds + lslot + ldso + j * 1024), 16, 0, 0); \
        lslot = (lslot + STAGE_BYTES == S * STAGE_BYTES) ? 0u : lslot + STAGE_BYTES; \
        if (++lk == NKT) { lk = 0; if (++lu == 1) { ubase = unit_base(1); rot = (idx & 3) * (KB1_ / 4) + (idx >> 2); kbm = KB1_ - 1; } } \
        else if (lk == NKT / 2 && gsrc >= 0) wait_x(1); } while (0)
#define WAITN(n) asm volatile("s_waitcnt vmcnt(%0)" :: "n"((n) * NLD) : "memory")
#pragma unroll
        for (int i = 0; i < S - 1; ++i) ISSUE();
        WAITN(S - 3);
        bar();
#pragma unroll 1
        for (int g = 0; g < G; ++g) {
            if (g + S - 1 < G) { ISSUE(); WAITN(S - 3); }
            else asm volatile("s_waitcnt vmcnt(0)" ::: "memory");
            bar();
        }
    }
}
}

extern "C" void kernel_launch(int rank, int nranks, ncclComm_t comm, hipStream_t stream,
                              void* const* d_in, const int* in_sizes, int n_in,
                              void* d_out, int out_size, const int* in_dtypes, int out_dtype,
                              void* const* d_out_all, void* const* d_ws_all, size_t ws_bytes) {
    if (nranks != NR || (unsigned)rank >= (unsigned)NR || ws_bytes < WS_XSTAGE + 2 * NR * XBLK_BYTES) { fprintf(stderr, "unexpected configuration\n"); return; }
    static bool init[16] = {false};
    int dev = 0; HIP_CHECK(hipGetDevice(&dev));
    if (!init[dev & 15]) {
        HIP_CHECK(hipFuncSetAttribute((const void*)fused_gemm_a2a, hipFuncAttributeMaxDynamicSharedMemorySize, LDS_BYTES));
        init[dev & 15] = true;
    }
    Params p;
    p.x = (const unsigned short*)d_in[0]; p.w = (const unsigned short*)d_in[1];
    for (int i = 0; i < 8; ++i) { p.out_all[i] = d_out_all[i]; p.ws_all[i] = d_ws_all[i]; }
    p.rank = rank;
    p.single = 0;
    hipLaunchKernelGGL(fused_gemm_a2a, dim3(NWG), dim3(NTHREADS), LDS_BYTES, stream, p);
    HIP_CHECK(hipGetLastError());
}
```

```cpp
#include <hip/hip_runtime.h>
#include <rccl/rccl.h>
#include <cstdio>

#define HIP_CHECK(expr) do { hipError_t _e = (expr); if (_e != hipSuccess) { \
    fprintf(stderr, "HIP error %s at %s:%d\n", hipGetErrorString(_e), __FILE__, __LINE__); \
    return; } } while (0)

namespace {
#define LAS __attribute__((address_space(3)))
typedef short bf16x8 __attribute__((ext_vector_type(8)));
typedef short s16x4 __attribute__((ext_vector_type(4)));
typedef float f32x4 __attribute__((ext_vector_type(4)));
typedef unsigned u32x4 __attribute__((ext_vector_type(4)));

#define NSTAGE_ 5
#define NX0_ 12
#define SEGY_ 6
#define SEGX1_ 5
#define SYNCT_ 4
#define KB0_ 4
#define KB1_ 4
constexpr int MLOC = 512, KD = 4096, ND = 16384, NR = 8, NPR = 2048;
constexpr int BM = 128, BN = 128, BK = 64, S = NSTAGE_;
constexpr int NKT = KD / BK;
constexpr int A_BYTES = BM * BK * 2, B_BYTES = BK * BN * 2, STAGE_BYTES = A_BYTES + B_BYTES;
constexpr int LDS_BYTES = S * STAGE_BYTES;
constexpr int NWG = 256;
constexpr int WM = 64, WN = 64, MI = 4, NI = 4;
constexpr int NLD = 8;
constexpr int NTHREADS = 8 * 64;
constexpr int BROW = BN * 2;
constexpr int G = 2 * NKT;
constexpr int NX0 = NX0_, NX1 = 14 - NX0_;
constexpr int WT = 17;
constexpr unsigned long long DEADLINE = 50000000ull;
static_assert(NX0 >= 11 && NX0 <= 14, "the lower-K half is flagged at slot 8, well before the end of unit 0");
static_assert(NX1 * SEGX1_ + 8 * SEGY_ + SYNCT_ < NKT, "slots must fit unit 1");

constexpr int WS_TMO = 64;
constexpr int WS_TICKET = 1024;
constexpr int WS_YFLAG = 4096;
constexpr int WS_XFLAG = 8192;
constexpr size_t WS_XSTAGE = 1u << 20;
constexpr size_t XBLK_BYTES = (size_t)256 * KD * 2;

struct Params {
    const unsigned short* x; const unsigned short* w;
    void* out_all[8]; void* ws_all[8];
    int rank; int single;
};

__device__ __forceinline__ void bar() { asm volatile("" ::: "memory"); __builtin_amdgcn_s_barrier(); asm volatile("" ::: "memory"); }
__device__ __forceinline__ unsigned cvt_pk_bf16(float lo, float hi) { unsigned r; asm("v_cvt_pk_bf16_f32 %0, %1, %2" : "=v"(r) : "v"(lo), "v"(hi)); return r; }
__device__ __forceinline__ float silu(float y) {
    const float c = __builtin_amdgcn_fmed3f(y, -60.f, 60.f);
    const float e = __builtin_amdgcn_exp2f(c * -1.44269504088896340736f);
    return y * __builtin_amdgcn_rcpf(1.0f + e);
}
__device__ __forceinline__ u32x4 pack8(const f32x4 va, const f32x4 vb) {
    const unsigned ax = cvt_pk_bf16(silu(va[0]), silu(va[1])), ay = cvt_pk_bf16(silu(va[2]), silu(va[3]));
    const unsigned bx = cvt_pk_bf16(silu(vb[0]), silu(vb[1])), by = cvt_pk_bf16(silu(vb[2]), silu(vb[3]));
    const auto sx = __builtin_amdgcn_permlane16_swap(ax, bx, false, false);
    const auto sy = __builtin_amdgcn_permlane16_swap(ay, by, false, false);
    return (u32x4){sx[0], sy[0], sx[1], sy[1]};
}

__device__ __forceinline__ void unit_coords(int wg, int ui, int rank, int& src, int& rb, int& dst, int& cb) {
    const int xcd = wg & 7, idx = wg >> 3;
    if (ui == 0) {
        const int u = xcd * 32 + idx, seq = u >> 1;
        src = rank; rb = u & 1;
        if (u < 224) { dst = (rank + 1 + seq % 7) & 7; cb = seq / 7; } else { dst = rank; cb = seq - 112; }
    } else {
        const int r = idx & 3;
        dst = rank; cb = (xcd & 1) * 8 + (idx >> 2);
        src = (rank + (xcd >> 1) * 2 + (r >> 1)) & 7; rb = 2 + (r & 1);
    }
}

#define TS(i) do {} while (0)

__global__ void __launch_bounds__(NTHREADS, 2) fused_gemm_a2a(const Params p) {
    extern __shared__ __attribute__((aligned(16))) unsigned char smem[];
    LAS unsigned char* lds = (LAS unsigned char*)smem;
    const int tid = threadIdx.x, wid = __builtin_amdgcn_readfirstlane(tid >> 6), lane = tid & 63;
    const int wg = blockIdx.x;
    unsigned* ws = (unsigned*)p.ws_all[p.rank];
    TS(0);

    if (wid < 4) {
        const int wr = wid >> 1, wc = wid & 1;
        const int fr = lane & 15, fq = lane >> 4;
        const unsigned ticket = __hip_atomic_load(ws + WS_TICKET + wg, __ATOMIC_RELAXED, __HIP_MEMORY_SCOPE_AGENT);
        const unsigned xo = (unsigned)(wg * 8192 + wid * 1024 + lane * 16);
        const u32x4 xv0 = *(const u32x4*)((const char*)p.x + XBLK_BYTES + xo), xv1 = *(const u32x4*)((const char*)p.x + XBLK_BYTES + xo + 4096);
        const unsigned long long tws = (unsigned long long)p.ws_all[lane & 7], tout = (unsigned long long)p.out_all[lane & 7];
        auto tab = [&](unsigned long long t, int r) {
            const unsigned lo = __builtin_amdgcn_readlane((unsigned)t, r & 7), hi = __builtin_amdgcn_readlane((unsigned)(t >> 32), r & 7);
            return (char*)(((unsigned long long)hi << 32) | lo);
        };
        const unsigned a0 = (unsigned)((wr * WM + fr) * 128 + ((fq ^ (fr >> 1)) * 16)), a1 = a0 ^ 64u;
        const int q = fr >> 2, pp = lane & 3, h = q | ((fq & 1) << 2);
        unsigned bo[NI];
#pragma unroll
        for (int ni = 0; ni < NI; ++ni) bo[ni] = (unsigned)(A_BYTES + (8 * fq + q) * BROW + (((wc * NI + ni) ^ h) * 32) + pp * 8);
        f32x4 ac0[MI][NI], ac1[MI][NI];
#pragma unroll
        for (int mi = 0; mi < MI; ++mi)
#pragma unroll
            for (int ni = 0; ni < NI; ++ni) { ac0[mi][ni] = (f32x4){0.f, 0.f, 0.f, 0.f}; ac1[mi][ni] = (f32x4){0.f, 0.f, 0.f, 0.f}; }
        bf16x8 af[2][MI], bf[2][NI];
#define LOADF(set, sbase, ks) do { \
        _Pragma("unroll") for (int mi = 0; mi < MI; ++mi) af[set][mi] = *(const LAS bf16x8*)(lds + (sbase) + ((ks) ? a1 : a0) + mi * 2048); \
        _Pragma("unroll") for (int ni = 0; ni < NI; ++ni) { \
            s16x4 lo = __builtin_amdgcn_ds_read_tr16_b64_v4i16((LAS s16x4*)(lds + (sbase) + bo[ni] + (ks) * 32 * BROW)); \
            s16x4 hi = __builtin_amdgcn_ds_read_tr16_b64_v4i16((LAS s16x4*)(lds + (sbase) + bo[ni] + (ks) * 32 * BROW + 4 * BROW)); \
            bf[set][ni] = __builtin_shufflevector(lo, hi, 0, 1, 2, 3, 4, 5, 6, 7); } } while (0)
#define MMA(acc, set) do { \
        _Pragma("unroll") for (int mi = 0; mi < MI; ++mi) _Pragma("unroll") for (int ni = 0; ni < NI; ++ni) \
            acc[mi][ni] = __builtin_amdgcn_mfma_f32_16x16x32_bf16(bf[set][ni], af[set][mi], acc[mi][ni], 0, 0, 0); } while (0)
#define INTERLEAVE() do { \
        _Pragma("unroll") for (int i = 0; i < MI + 2 * NI; ++i) { __builtin_amdgcn_sched_group_barrier(0x8, 1, 0); __builtin_amdgcn_sched_group_barrier(0x100, 1, 0); } \
        __builtin_amdgcn_sched_group_barrier(0x8, MI * NI - (MI + 2 * NI), 0); } while (0)
#define TILE(acc) do { \
            const unsigned sn = (sb + STAGE_BYTES == S * STAGE_BYTES) ? 0u : sb + STAGE_BYTES; \
            LOADF(1, sb, 1); MMA(acc, 0); INTERLEAVE(); \
            LOADF(0, sn, 0); MMA(acc, 1); INTERLEAVE(); \
            sb = sn; } while (0)
#define INTERLEAVE_V() do { \
        _Pragma("unroll") for (int i = 0; i < MI + 2 * NI; ++i) { __builtin_amdgcn_sched_group_barrier(0x8, 1, 0); __builtin_amdgcn_sched_group_barrier(0x100, 1, 0); __builtin_amdgcn_sched_group_barrier(0x402, 2, 0); } \
        _Pragma("unroll") for (int i = 0; i < MI * NI - (MI + 2 * NI); ++i) { __builtin_amdgcn_sched_group_barrier(0x8, 1, 0); __builtin_amdgcn_sched_group_barrier(0x402, 3, 0); } } while (0)
        bar();
        TS(1);
        LOADF(0, 0u, 0);
        unsigned sb = 0;
        unsigned call = 0;
        auto push_x = [&](int sl) {
            const int pr = (p.rank + 1 + (sl + wg + wid) % 7) & 7;
            const __amdgpu_buffer_rsrc_t xr = __builtin_amdgcn_make_buffer_rsrc(tab(tws, pr) + WS_XSTAGE + ((size_t)(call & 1) * NR + p.rank) * XBLK_BYTES, 0, (int)XBLK_BYTES, 0x00020000);
            if (sl < 7) __builtin_amdgcn_raw_buffer_store_b128(xv0, xr, xo, 0, WT);
            else __builtin_amdgcn_raw_buffer_store_b128(xv1, xr, xo + 4096, 0, WT);
        };
        auto flag_x = [&](int kh) {
            if (wid == 0 && lane < 8 && lane != p.rank && !p.single)
                __hip_atomic_store((unsigned*)tws + WS_XFLAG + 512 * p.rank + 256 * kh + wg, call, __ATOMIC_RELAXED, __HIP_MEMORY_SCOPE_SYSTEM);
        };
        {
            int done = 0;
#pragma unroll 1
            for (int sl = 0; sl < NX0; ++sl) {
                const int nt = ((sl + 1) * NKT) / NX0 - done;
                done += nt;
#pragma unroll 1
                for (int t = 0; t < nt; ++t) {
                    TILE(ac0);
                    if (t == nt - 1) {
                        if (sl == 0) call = (unsigned)__builtin_amdgcn_readfirstlane(ticket) + 1u;
                        if (sl == 1 && tid == 0) __hip_atomic_store(ws + WS_TICKET + wg, call, __ATOMIC_RELAXED, __HIP_MEMORY_SCOPE_AGENT);
                        if (sl == 8) asm volatile("s_waitcnt vmcnt(1)" ::: "memory");
                        if (!p.single) push_x(sl);
                    }
                    bar();
                }
                if (sl == 8) flag_x(0);
                if (sl == NX0 - 2) TS(9);
            }
        }
        TS(2); TS(3);
        int src, rb, dst, cb; unit_coords(wg, 0, p.rank, src, rb, dst, cb);
        dst = __builtin_amdgcn_readfirstlane(dst) & 7;
        if (p.single) dst = p.rank;
        const unsigned lofs = (unsigned)(((wr * WM + fr) * NPR + wc * WN + (fq & 1) * 16 + (fq >> 1) * 8) * 2);
        unsigned tmo;
        {
            const __amdgpu_buffer_rsrc_t rs = __builtin_amdgcn_make_buffer_rsrc(tab(tout, dst), 0, 4096 * NPR * 2, 0x00020000);
            const unsigned gofs = (unsigned)(((p.rank * MLOC + rb * BM) * NPR + cb * BN) * 2) + lofs;
#pragma unroll 1
            for (int sl = NX0; sl < 14; ++sl) {
#pragma unroll 1
                for (int t = 0; t < SEGX1_; ++t) {
                    TILE(ac1);
                    if (t == SEGX1_ - 1 && !p.single) push_x(sl);
                    bar();
                }
            }
#pragma unroll
            for (int seg = 0; seg < 8; ++seg) {
#pragma unroll 1
                for (int t = 0; t < SEGY_ - 1; ++t) { TILE(ac1); bar(); }
                {
                    const unsigned sn = (sb + STAGE_BYTES == S * STAGE_BYTES) ? 0u : sb + STAGE_BYTES;
                    const u32x4 o = pack8(ac0[seg >> 1][2 * (seg & 1)], ac0[seg >> 1][2 * (seg & 1) + 1]);
                    LOADF(1, sb, 1); MMA(ac1, 0); INTERLEAVE_V();
                    LOADF(0, sn, 0); MMA(ac1, 1); INTERLEAVE_V();
                    sb = sn;
                    if (seg == 0) asm volatile("s_waitcnt vmcnt(0)" ::: "memory");
                    __builtin_amdgcn_raw_buffer_store_b128(o, rs, gofs + (unsigned)(((seg >> 1) * 16 * NPR + (seg & 1) * 32) * 2), 0, WT);
                    bar();
                }
                if (seg == 0) flag_x(1);
            }
#pragma unroll 1
            for (int t = 0; t < SYNCT_ - 1; ++t) { TILE(ac1); bar(); }
            TILE(ac1);
            asm volatile("s_waitcnt vmcnt(0)" ::: "memory");
            if (dst != p.rank && lane == dst)
                __hip_atomic_store((unsigned*)tws + WS_YFLAG + 256 * p.rank + (cb * 2 + rb) * 4 + wid, call, __ATOMIC_RELAXED, __HIP_MEMORY_SCOPE_SYSTEM);
            tmo = __hip_atomic_load(ws + WS_TMO, __ATOMIC_RELAXED, __HIP_MEMORY_SCOPE_AGENT);
            bar();
#pragma unroll 1
            for (int t = 0; t < NKT - NX1 * SEGX1_ - 8 * SEGY_ - SYNCT_; ++t) { TILE(ac1); bar(); }
        }
        TS(4);
        {
            unit_coords(wg, 1, p.rank, src, rb, dst, cb);
            src = __builtin_amdgcn_readfirstlane(src) & 7;
            const bool bad = src != p.rank && (unsigned)__builtin_amdgcn_readfirstlane(tmo) == call;
            const __amdgpu_buffer_rsrc_t rs = __builtin_amdgcn_make_buffer_rsrc(p.out_all[p.rank], 0, 4096 * NPR * 2, 0x00020000);
            const unsigned gofs = (unsigned)(((src * MLOC + rb * BM) * NPR + cb * BN) * 2) + lofs;
#pragma unroll
            for (int i = 0; i < 8; ++i) {
                u32x4 o = pack8(ac1[i >> 1][2 * (i & 1)], ac1[i >> 1][2 * (i & 1) + 1]);
                if (bad) o = (u32x4){0x7fc07fc0u, 0x7fc07fc0u, 0x7fc07fc0u, 0x7fc07fc0u};
                __builtin_amdgcn_raw_buffer_store_b128(o, rs, gofs + (unsigned)(((i >> 1) * 16 * NPR + (i & 1) * 32) * 2), 0, WT);
            }
        }
        TS(14);
        if (wg < 7 * 8 && (wg & 7) == 0 && wid == 0 && !p.single) {
            const unsigned* fp = ws + WS_YFLAG + 256 * ((p.rank + 1 + (wg >> 3)) & 7) + lane * 2;
            const unsigned long long t0 = __builtin_amdgcn_s_memrealtime();
            bool to = false;
            for (;;) {
                const unsigned f0 = __hip_atomic_load(fp, __ATOMIC_RELAXED, __HIP_MEMORY_SCOPE_SYSTEM), f1 = __hip_atomic_load(fp + 1, __ATOMIC_RELAXED, __HIP_MEMORY_SCOPE_SYSTEM);
                if (__all(f0 >= call && f1 >= call)) break;
                if ((unsigned)__builtin_amdgcn_readfirstlane(__hip_atomic_load(ws + WS_TMO, __ATOMIC_RELAXED, __HIP_MEMORY_SCOPE_AGENT)) == call
                    || __builtin_amdgcn_s_memrealtime() - t0 > DEADLINE) { to = true; break; }
                __builtin_amdgcn_s_sleep(1);
            }
            __builtin_amdgcn_fence(__ATOMIC_ACQUIRE, "");
            if (to) {
                asm volatile("s_waitcnt vmcnt(0)" ::: "memory");
                if (lane == 0) {
                    __hip_atomic_store(ws + WS_TMO, call, __ATOMIC_RELAXED, __HIP_MEMORY_SCOPE_AGENT);
                    ((unsigned short*)p.out_all[p.rank])[((size_t)(src * MLOC + rb * BM)) * NPR + cb * BN] = 0x7fc0;
                }
            }
        }
        TS(15);
    } else {
        const int li = wid - 4;
        const bool isA = li < 2;
        const int half = li & 1;
        unsigned voff[NLD];
        if (isA) {
#pragma unroll
            for (int j = 0; j < NLD; ++j) { const int row = (half * NLD + j) * 8 + (lane >> 3); voff[j] = (unsigned)(row * KD * 2 + (((lane & 7) ^ ((row >> 1) & 7)) * 16)); }
        } else {
#pragma unroll
            for (int j = 0; j < NLD; ++j) { const int kr = (half * NLD + j) * 4 + (lane >> 4); const int hh = (kr & 3) | (((kr >> 3) & 1) << 2);
                voff[j] = (unsigned)(kr * ND * 2 + (((lane & 15) ^ (hh << 1)) * 16)); }
        }
        const unsigned ldso = isA ? (unsigned)(half * NLD * 1024) : (unsigned)(A_BYTES + half * NLD * 1024);
        const size_t kstep = isA ? (size_t)BK * 2 : (size_t)BK * ND * 2;
        int lu = 0, lk = 0; unsigned lslot = 0;
        int gsrc = -1; unsigned call = 0;
        auto wait_x = [&](int kh) {
            const unsigned long long t0 = __builtin_amdgcn_s_memrealtime();
            const unsigned* fp = ws + WS_XFLAG + 512 * gsrc + 256 * kh + lane * 4;
            for (;;) {
                const unsigned f0 = __hip_atomic_load(fp, __ATOMIC_RELAXED, __HIP_MEMORY_SCOPE_SYSTEM), f1 = __hip_atomic_load(fp + 1, __ATOMIC_RELAXED, __HIP_MEMORY_SCOPE_SYSTEM),
                               f2 = __hip_atomic_load(fp + 2, __ATOMIC_RELAXED, __HIP_MEMORY_SCOPE_SYSTEM), f3 = __hip_atomic_load(fp + 3, __ATOMIC_RELAXED, __HIP_MEMORY_SCOPE_SYSTEM);
                if (__all(f0 >= call && f1 >= call && f2 >= call && f3 >= call)) break;
                if ((unsigned)__builtin_amdgcn_readfirstlane(__hip_atomic_load(ws + WS_TMO, __ATOMIC_RELAXED, __HIP_MEMORY_SCOPE_AGENT)) == call
                    || __builtin_amdgcn_s_memrealtime() - t0 > DEADLINE) {
                    __hip_atomic_store(ws + WS_TMO, call, __ATOMIC_RELAXED, __HIP_MEMORY_SCOPE_AGENT);
                    break;
                }
                __builtin_amdgcn_s_sleep(2);
            }
            __builtin_amdgcn_fence(__ATOMIC_ACQUIRE, "wavefront");
        };
        auto unit_base = [&](int ui) {
            int src, rb, dst, cb; unit_coords(wg, ui, p.rank, src, rb, dst, cb);
            gsrc = -1;
            if (!isA) return (const char*)p.w + ((size_t)dst * NPR + (size_t)cb * BN) * 2;
            if (src == p.rank || p.single) return (const char*)p.x + (size_t)rb * BM * KD * 2;
            call = (unsigned)__builtin_amdgcn_readfirstlane(__hip_atomic_load(ws + WS_TICKET + wg, __ATOMIC_RELAXED, __HIP_MEMORY_SCOPE_AGENT));
            gsrc = src; wait_x(0);
            return (const char*)ws + WS_XSTAGE + ((size_t)(call & 1) * NR + src) * XBLK_BYTES + (size_t)(rb - 2) * BM * KD * 2;
        };
        const int idx = wg >> 3;
        int rot = (idx & 1) * (KB0_ / 2) + (idx >> 1), kbm = KB0_ - 1;
        const char* ubase = unit_base(0);
#define ISSUE() do { \
        const char* base = ubase + (size_t)((lk & ~kbm) + ((lk + rot) & kbm)) * kstep; \
        _Pragma("unroll") for (int j = 0; j < NLD; ++j) __builtin_amdgcn_global_load_lds((const unsigned*)(base + voff[j]), (LAS unsigned*)(lds + lslot + ldso + j * 1024), 16, 0, 0); \
        lslot = (lslot + STAGE_BYTES == S * STAGE_BYTES) ? 0u : lslot + STAGE_BYTES; \
        if (++lk == NKT) { lk = 0; if (++lu == 1) { ubase = unit_base(1); rot = (idx & 3) * (KB1_ / 4) + (idx >> 2); kbm = KB1_ - 1; } } \
        else if (lk == NKT / 2 && gsrc >= 0) wait_x(1); } while (0)
#define WAITN(n) asm volatile("s_waitcnt vmcnt(%0)" :: "n"((n) * NLD) : "memory")
#pragma unroll
        for (int i = 0; i < S - 1; ++i) ISSUE();
        WAITN(S - 3);
        bar();
#pragma unroll 1
        for (int g = 0; g < G; ++g) {
            if (g + S - 1 < G) { ISSUE(); WAITN(S - 3); }
            else asm volatile("s_waitcnt vmcnt(0)" ::: "memory");
            bar();
        }
    }
}
}

extern "C" void kernel_launch(int rank, int nranks, ncclComm_t comm, hipStream_t stream,
                              void* const* d_in, const int* in_sizes, int n_in,
                              void* d_out, int out_size, const int* in_dtypes, int out_dtype,
                              void* const* d_out_all, void* const* d_ws_all, size_t ws_bytes) {
    if (nranks != NR || (unsigned)rank >= (unsigned)NR || ws_bytes < WS_XSTAGE + 2 * NR * XBLK_BYTES) { fprintf(stderr, "unexpected configuration\n"); return; }
    static bool init[16] = {false};
    int dev = 0; HIP_CHECK(hipGetDevice(&dev));
    if (!init[dev & 15]) {
        HIP_CHECK(hipFuncSetAttribute((const void*)fused_gemm_a2a, hipFuncAttributeMaxDynamicSharedMemorySize, LDS_BYTES));
        init[dev & 15] = true;
    }
    Params p;
    p.x = (const unsigned short*)d_in[0]; p.w = (const unsigned short*)d_in[1];
    for (int i = 0; i < 8; ++i) { p.out_all[i] = d_out_all[i]; p.ws_all[i] = d_ws_all[i]; }
    p.rank = rank;
    p.single = 0;
    hipLaunchKernelGGL(fused_gemm_a2a, dim3(NWG), dim3(NTHREADS), LDS_BYTES, stream, p);
    HIP_CHECK(hipGetLastError());
}
```

```cpp
#include <hip/hip_runtime.h>
#include <rccl/rccl.h>
#include <cstdio>

#define HIP_CHECK(expr) do { hipError_t _e = (expr); if (_e != hipSuccess) { \
    fprintf(stderr, "HIP error %s at %s:%d\n", hipGetErrorString(_e), __FILE__, __LINE__); \
    return; } } while (0)

namespace {
#define LAS __attribute__((address_space(3)))
typedef short bf16x8 __attribute__((ext_vector_type(8)));
typedef short s16x4 __attribute__((ext_vector_type(4)));
typedef float f32x4 __attribute__((ext_vector_type(4)));
typedef unsigned u32x4 __attribute__((ext_vector_type(4)));

#define NSTAGE_ 5
#define NX0_ 13
#define SEGY_ 5
#define SEGX1_ 5
#define SYNCT_ 3
#define KB0_ 4
#define KB1_ 4
constexpr int MLOC = 512, KD = 4096, ND = 16384, NR = 8, NPR = 2048;
constexpr int BM = 128, BN = 128, BK = 64, S = NSTAGE_;
constexpr int NKT = KD / BK;
constexpr int A_BYTES = BM * BK * 2, B_BYTES = BK * BN * 2, STAGE_BYTES = A_BYTES + B_BYTES;
constexpr int LDS_BYTES = S * STAGE_BYTES;
constexpr int NWG = 256;
constexpr int WM = 64, WN = 64, MI = 4, NI = 4;
constexpr int NLD = 8;
constexpr int NTHREADS = 8 * 64;
constexpr int BROW = BN * 2;
constexpr int G = 2 * NKT;
constexpr int NX0 = NX0_, NX1 = 14 - NX0_;
constexpr int WT = 17;
constexpr unsigned long long DEADLINE = 50000000ull;
static_assert(NX0 >= 11 && NX0 <= 14, "the lower-K half is flagged at slot 8, well before the end of unit 0");
static_assert(NX1 * SEGX1_ + 8 * SEGY_ + SYNCT_ < NKT && NX1 * SEGX1_ + 8 * SEGY_ + SYNCT_ > NKT / 2, "slots must fit unit 1; the timeout word is read after the loaders' last wait");
static_assert(NX1 * SEGX1_ + SEGY_ < NKT / 2 - 8, "the upper-K half is flagged well before the peers' loaders wait for it");

constexpr int WS_TMO = 64;
constexpr int WS_TICKET = 1024;
constexpr int WS_YFLAG = 4096;
constexpr int WS_XFLAG = 8192;
constexpr size_t WS_XSTAGE = 1u << 20;
constexpr size_t XBLK_BYTES = (size_t)256 * KD * 2;

struct Params {
    const unsigned short* x; const unsigned short* w;
    void* out_all[8]; void* ws_all[8];
    int rank; int single;
};

__device__ __forceinline__ void bar() { asm volatile("" ::: "memory"); __builtin_amdgcn_s_barrier(); asm volatile("" ::: "memory"); }
__device__ __forceinline__ unsigned cvt_pk_bf16(float lo, float hi) { unsigned r; asm("v_cvt_pk_bf16_f32 %0, %1, %2" : "=v"(r) : "v"(lo), "v"(hi)); return r; }
__device__ __forceinline__ float silu(float y) {
    const float c = __builtin_amdgcn_fmed3f(y, -60.f, 60.f);
    const float e = __builtin_amdgcn_exp2f(c * -1.44269504088896340736f);
    return y * __builtin_amdgcn_rcpf(1.0f + e);
}
__device__ __forceinline__ u32x4 pack8(const f32x4 va, const f32x4 vb) {
    const unsigned ax = cvt_pk_bf16(silu(va[0]), silu(va[1])), ay = cvt_pk_bf16(silu(va[2]), silu(va[3]));
    const unsigned bx = cvt_pk_bf16(silu(vb[0]), silu(vb[1])), by = cvt_pk_bf16(silu(vb[2]), silu(vb[3]));
    const auto sx = __builtin_amdgcn_permlane16_swap(ax, bx, false, false);
    const auto sy = __builtin_amdgcn_permlane16_swap(ay, by, false, false);
    return (u32x4){sx[0], sy[0], sx[1], sy[1]};
}

__device__ __forceinline__ void unit_coords(int wg, int ui, int rank, int& src, int& rb, int& dst, int& cb) {
    const int xcd = wg & 7, idx = wg >> 3;
    if (ui == 0) {
        const int u = xcd * 32 + idx, seq = u >> 1;
        src = rank; rb = u & 1;
        if (u < 224) { dst = (rank + 1 + seq % 7) & 7; cb = seq / 7; } else { dst = rank; cb = seq - 112; }
    } else {
        const int r = idx & 3;
        dst = rank; cb = (xcd & 1) * 8 + (idx >> 2);
        src = (rank + (xcd >> 1) * 2 + (r >> 1)) & 7; rb = 2 + (r & 1);
    }
}

#define TS(i) do {} while (0)

__global__ void __launch_bounds__(NTHREADS, 2) fused_gemm_a2a(const Params p) {
    extern __shared__ __attribute__((aligned(16))) unsigned char smem[];
    LAS unsigned char* lds = (LAS unsigned char*)smem;
    const int tid = threadIdx.x, wid = __builtin_amdgcn_readfirstlane(tid >> 6), lane = tid & 63;
    const int wg = blockIdx.x;
    unsigned* ws = (unsigned*)p.ws_all[p.rank];
    TS(0);

    if (wid < 4) {
        const int wr = wid >> 1, wc = wid & 1;
        const int fr = lane & 15, fq = lane >> 4;
        const unsigned ticket = __hip_atomic_load(ws + WS_TICKET + wg, __ATOMIC_RELAXED, __HIP_MEMORY_SCOPE_AGENT);
        const unsigned xo = (unsigned)(wg * 8192 + wid * 1024 + lane * 16);
        const u32x4 xv0 = *(const u32x4*)((const char*)p.x + XBLK_BYTES + xo), xv1 = *(const u32x4*)((const char*)p.x + XBLK_BYTES + xo + 4096);
        const unsigned long long tws = (unsigned long long)p.ws_all[lane & 7], tout = (unsigned long long)p.out_all[lane & 7];
        auto tab = [&](unsigned long long t, int r) {
            const unsigned lo = __builtin_amdgcn_readlane((unsigned)t, r & 7), hi = __builtin_amdgcn_readlane((unsigned)(t >> 32), r & 7);
            return (char*)(((unsigned long long)hi << 32) | lo);
        };
        const unsigned a0 = (unsigned)((wr * WM + fr) * 128 + ((fq ^ (fr >> 1)) * 16)), a1 = a0 ^ 64u;
        const int q = fr >> 2, pp = lane & 3, h = q | ((fq & 1) << 2);
        unsigned bo[NI];
#pragma unroll
        for (int ni = 0; ni < NI; ++ni) bo[ni] = (unsigned)(A_BYTES + (8 * fq + q) * BROW + (((wc * NI + ni) ^ h) * 32) + pp * 8);
        f32x4 ac0[MI][NI], ac1[MI][NI];
#pragma unroll
        for (int mi = 0; mi < MI; ++mi)
#pragma unroll
            for (int ni = 0; ni < NI; ++ni) { ac0[mi][ni] = (f32x4){0.f, 0.f, 0.f, 0.f}; ac1[mi][ni] = (f32x4){0.f, 0.f, 0.f, 0.f}; }
        bf16x8 af[2][MI], bf[2][NI];
#define LOADF(set, sbase, ks) do { \
        _Pragma("unroll") for (int mi = 0; mi < MI; ++mi) af[set][mi] = *(const LAS bf16x8*)(lds + (sbase) + ((ks) ? a1 : a0) + mi * 2048); \
        _Pragma("unroll") for (int ni = 0; ni < NI; ++ni) { \
            s16x4 lo = __builtin_amdgcn_ds_read_tr16_b64_v4i16((LAS s16x4*)(lds + (sbase) + bo[ni] + (ks) * 32 * BROW)); \
            s16x4 hi = __builtin_amdgcn_ds_read_tr16_b64_v4i16((LAS s16x4*)(lds + (sbase) + bo[ni] + (ks) * 32 * BROW + 4 * BROW)); \
            bf[set][ni] = __builtin_shufflevector(lo, hi, 0, 1, 2, 3, 4, 5, 6, 7); } } while (0)
#define MMA(acc, set) do { \
        _Pragma("unroll") for (int mi = 0; mi < MI; ++mi) _Pragma("unroll") for (int ni = 0; ni < NI; ++ni) \
            acc[mi][ni] = __builtin_amdgcn_mfma_f32_16x16x32_bf16(bf[set][ni], af[set][mi], acc[mi][ni], 0, 0, 0); } while (0)
#define INTERLEAVE() do { \
        _Pragma("unroll") for (int i = 0; i < MI + 2 * NI; ++i) { __builtin_amdgcn_sched_group_barrier(0x8, 1, 0); __builtin_amdgcn_sched_group_barrier(0x100, 1, 0); } \
        __builtin_amdgcn_sched_group_barrier(0x8, MI * NI - (MI + 2 * NI), 0); } while (0)
#define TILE(acc) do { \
            const unsigned sn = (sb + STAGE_BYTES == S * STAGE_BYTES) ? 0u : sb + STAGE_BYTES; \
            LOADF(1, sb, 1); MMA(acc, 0); INTERLEAVE(); \
            LOADF(0, sn, 0); MMA(acc, 1); INTERLEAVE(); \
            sb = sn; } while (0)
#define INTERLEAVE_V() do { \
        _Pragma("unroll") for (int i = 0; i < MI + 2 * NI; ++i) { __builtin_amdgcn_sched_group_barrier(0x8, 1, 0); __builtin_amdgcn_sched_group_barrier(0x100, 1, 0); __builtin_amdgcn_sched_group_barrier(0x402, 2, 0); } \
        _Pragma("unroll") for (int i = 0; i < MI * NI - (MI + 2 * NI); ++i) { __builtin_amdgcn_sched_group_barrier(0x8, 1, 0); __builtin_amdgcn_sched_group_barrier(0x402, 3, 0); } } while (0)
        bar();
        TS(1);
        LOADF(0, 0u, 0);
        unsigned sb = 0;
        unsigned call = 0;
        auto push_x = [&](int sl) {
            const int pr = (p.rank + 1 + (sl + wg + wid) % 7) & 7;
            const __amdgpu_buffer_rsrc_t xr = __builtin_amdgcn_make_buffer_rsrc(tab(tws, pr) + WS_XSTAGE + ((size_t)(call & 1) * NR + p.rank) * XBLK_BYTES, 0, (int)XBLK_BYTES, 0x00020000);
            if (sl < 7) __builtin_amdgcn_raw_buffer_store_b128(xv0, xr, xo, 0, WT);
            else __builtin_amdgcn_raw_buffer_store_b128(xv1, xr, xo + 4096, 0, WT);
        };
        auto flag_x = [&](int kh) {
            if (wid == 0 && lane < 8 && lane != p.rank && !p.single)
                __hip_atomic_store((unsigned*)tws + WS_XFLAG + 512 * p.rank + 256 * kh + wg, call, __ATOMIC_RELAXED, __HIP_MEMORY_SCOPE_SYSTEM);
        };
        {
            int done = 0;
#pragma unroll 1
            for (int sl = 0; sl < NX0; ++sl) {
                const int nt = ((sl + 1) * NKT) / NX0 - done;
                done += nt;
#pragma unroll 1
                for (int t = 0; t < nt; ++t) {
                    TILE(ac0);
                    if (t == nt - 1) {
                        if (sl == 0) call = (unsigned)__builtin_amdgcn_readfirstlane(ticket) + 1u;
                        if (sl == 1 && tid == 0) __hip_atomic_store(ws + WS_TICKET + wg, call, __ATOMIC_RELAXED, __HIP_MEMORY_SCOPE_AGENT);
                        if (sl == 8) asm volatile("s_waitcnt vmcnt(1)" ::: "memory");
                        if (!p.single) push_x(sl);
                    }
                    bar();
                }
                if (sl == 8) flag_x(0);
                if (sl == NX0 - 2) TS(9);
            }
        }
        TS(2); TS(3);
        int src, rb, dst, cb; unit_coords(wg, 0, p.rank, src, rb, dst, cb);
        dst = __builtin_amdgcn_readfirstlane(dst) & 7;
        if (p.single) dst = p.rank;
        const unsigned lofs = (unsigned)(((wr * WM + fr) * NPR + wc * WN + (fq & 1) * 16 + (fq >> 1) * 8) * 2);
        unsigned tmo;
        {
            const __amdgpu_buffer_rsrc_t rs = __builtin_amdgcn_make_buffer_rsrc(tab(tout, dst), 0, 4096 * NPR * 2, 0x00020000);
            const unsigned gofs = (unsigned)(((p.rank * MLOC + rb * BM) * NPR + cb * BN) * 2) + lofs;
#pragma unroll 1
            for (int sl = NX0; sl < 14; ++sl) {
#pragma unroll 1
                for (int t = 0; t < SEGX1_; ++t) {
                    TILE(ac1);
                    if (t == SEGX1_ - 1 && !p.single) push_x(sl);
                    bar();
                }
            }
#pragma unroll
            for (int seg = 0; seg < 8; ++seg) {
#pragma unroll 1
                for (int t = 0; t < SEGY_ - 1; ++t) { TILE(ac1); bar(); }
                {
                    const unsigned sn = (sb + STAGE_BYTES == S * STAGE_BYTES) ? 0u : sb + STAGE_BYTES;
                    const u32x4 o = pack8(ac0[seg >> 1][2 * (seg & 1)], ac0[seg >> 1][2 * (seg & 1) + 1]);
                    LOADF(1, sb, 1); MMA(ac1, 0); INTERLEAVE_V();
                    LOADF(0, sn, 0); MMA(ac1, 1); INTERLEAVE_V();
                    sb = sn;
                    if (seg == 0) asm volatile("s_waitcnt vmcnt(0)" ::: "memory");
                    __builtin_amdgcn_raw_buffer_store_b128(o, rs, gofs + (unsigned)(((seg >> 1) * 16 * NPR + (seg & 1) * 32) * 2), 0, WT);
                    bar();
                }
                if (seg == 0) flag_x(1);
            }
#pragma unroll 1
            for (int t = 0; t < SYNCT_ - 1; ++t) { TILE(ac1); bar(); }
            TILE(ac1);
            asm volatile("s_waitcnt vmcnt(0)" ::: "memory");
            if (dst != p.rank && lane == dst)
                __hip_atomic_store((unsigned*)tws + WS_YFLAG + 256 * p.rank + (cb * 2 + rb) * 4 + wid, call, __ATOMIC_RELAXED, __HIP_MEMORY_SCOPE_SYSTEM);
            tmo = __hip_atomic_load(ws + WS_TMO, __ATOMIC_RELAXED, __HIP_MEMORY_SCOPE_AGENT);
            bar();
#pragma unroll 1
            for (int t = 0; t < NKT - NX1 * SEGX1_ - 8 * SEGY_ - SYNCT_; ++t) { TILE(ac1); bar(); }
        }
        TS(4);
        {
            unit_coords(wg, 1, p.rank, src, rb, dst, cb);
            src = __builtin_amdgcn_readfirstlane(src) & 7;
            const bool bad = src != p.rank && (unsigned)__builtin_amdgcn_readfirstlane(tmo) == call;
            const __amdgpu_buffer_rsrc_t rs = __builtin_amdgcn_make_buffer_rsrc(p.out_all[p.rank], 0, 4096 * NPR * 2, 0x00020000);
            const unsigned gofs = (unsigned)(((src * MLOC + rb * BM) * NPR + cb * BN) * 2) + lofs;
#pragma unroll
            for (int i = 0; i < 8; ++i) {
                u32x4 o = pack8(ac1[i >> 1][2 * (i & 1)], ac1[i >> 1][2 * (i & 1) + 1]);
                if (bad) o = (u32x4){0x7fc07fc0u, 0x7fc07fc0u, 0x7fc07fc0u, 0x7fc07fc0u};
                __builtin_amdgcn_raw_buffer_store_b128(o, rs, gofs + (unsigned)(((i >> 1) * 16 * NPR + (i & 1) * 32) * 2), 0, WT);
            }
        }
        TS(14);
        if (wg < 7 * 8 && (wg & 7) == 0 && wid == 0 && !p.single) {
            const unsigned* fp = ws + WS_YFLAG + 256 * ((p.rank + 1 + (wg >> 3)) & 7) + lane * 2;
            const unsigned long long t0 = __builtin_amdgcn_s_memrealtime();
            bool to = false;
            for (;;) {
                const unsigned f0 = __hip_atomic_load(fp, __ATOMIC_RELAXED, __HIP_MEMORY_SCOPE_SYSTEM), f1 = __hip_atomic_load(fp + 1, __ATOMIC_RELAXED, __HIP_MEMORY_SCOPE_SYSTEM);
                if (__all(f0 >= call && f1 >= call)) break;
                if ((unsigned)__builtin_amdgcn_readfirstlane(__hip_atomic_load(ws + WS_TMO, __ATOMIC_RELAXED, __HIP_MEMORY_SCOPE_AGENT)) == call
                    || __builtin_amdgcn_s_memrealtime() - t0 > DEADLINE) { to = true; break; }
                __builtin_amdgcn_s_sleep(1);
            }
            __builtin_amdgcn_fence(__ATOMIC_ACQUIRE, "");
            if (to) {
                asm volatile("s_waitcnt vmcnt(0)" ::: "memory");
                if (lane == 0) {
                    __hip_atomic_store(ws + WS_TMO, call, __ATOMIC_RELAXED, __HIP_MEMORY_SCOPE_AGENT);
                    ((unsigned short*)p.out_all[p.rank])[((size_t)(src * MLOC + rb * BM)) * NPR + cb * BN] = 0x7fc0;
                }
            }
        }
        TS(15);
    } else {
        const int li = wid - 4;
        const bool isA = li < 2;
        const int half = li & 1;
        unsigned voff[NLD];
        if (isA) {
#pragma unroll
            for (int j = 0; j < NLD; ++j) { const int row = (half * NLD + j) * 8 + (lane >> 3); voff[j] = (unsigned)(row * KD * 2 + (((lane & 7) ^ ((row >> 1) & 7)) * 16)); }
        } else {
#pragma unroll
            for (int j = 0; j < NLD; ++j) { const int kr = (half * NLD + j) * 4 + (lane >> 4); const int hh = (kr & 3) | (((kr >> 3) & 1) << 2);
                voff[j] = (unsigned)(kr * ND * 2 + (((lane & 15) ^ (hh << 1)) * 16)); }
        }
        const unsigned ldso = isA ? (unsigned)(half * NLD * 1024) : (unsigned)(A_BYTES + half * NLD * 1024);
        const size_t kstep = isA ? (size_t)BK * 2 : (size_t)BK * ND * 2;
        int lu = 0, lk = 0; unsigned lslot = 0;
        int gsrc = -1; unsigned call = 0;
        auto wait_x = [&](int kh) {
            const unsigned long long t0 = __builtin_amdgcn_s_memrealtime();
            const unsigned* fp = ws + WS_XFLAG + 512 * gsrc + 256 * kh + lane * 4;
            for (;;) {
                const unsigned f0 = __hip_atomic_load(fp, __ATOMIC_RELAXED, __HIP_MEMORY_SCOPE_SYSTEM), f1 = __hip_atomic_load(fp + 1, __ATOMIC_RELAXED, __HIP_MEMORY_SCOPE_SYSTEM),
                               f2 = __hip_atomic_load(fp + 2, __ATOMIC_RELAXED, __HIP_MEMORY_SCOPE_SYSTEM), f3 = __hip_atomic_load(fp + 3, __ATOMIC_RELAXED, __HIP_MEMORY_SCOPE_SYSTEM);
                if (__all(f0 >= call && f1 >= call && f2 >= call && f3 >= call)) break;
                if ((unsigned)__builtin_amdgcn_readfirstlane(__hip_atomic_load(ws + WS_TMO, __ATOMIC_RELAXED, __HIP_MEMORY_SCOPE_AGENT)) == call
                    || __builtin_amdgcn_s_memrealtime() - t0 > DEADLINE) {
                    __hip_atomic_store(ws + WS_TMO, call, __ATOMIC_RELAXED, __HIP_MEMORY_SCOPE_AGENT);
                    break;
                }
                __builtin_amdgcn_s_sleep(2);
            }
            __builtin_amdgcn_fence(__ATOMIC_ACQUIRE, "wavefront");
        };
        auto unit_base = [&](int ui) {
            int src, rb, dst, cb; unit_coords(wg, ui, p.rank, src, rb, dst, cb);
            gsrc = -1;
            if (!isA) return (const char*)p.w + ((size_t)dst * NPR + (size_t)cb * BN) * 2;
            if (src == p.rank || p.single) return (const char*)p.x + (size_t)rb * BM * KD * 2;
            call = (unsigned)__builtin_amdgcn_readfirstlane(__hip_atomic_load(ws + WS_TICKET + wg, __ATOMIC_RELAXED, __HIP_MEMORY_SCOPE_AGENT));
            gsrc = src; wait_x(0);
            return (const char*)ws + WS_XSTAGE + ((size_t)(call & 1) * NR + src) * XBLK_BYTES + (size_t)(rb - 2) * BM * KD * 2;
        };
        const int idx = wg >> 3;
        int rot = (idx & 1) * (KB0_ / 2) + (idx >> 1), kbm = KB0_ - 1;
        const char* ubase = unit_base(0);
#define ISSUE() do { \
        const char* base = ubase + (size_t)((lk & ~kbm) + ((lk + rot) & kbm)) * kstep; \
        _Pragma("unroll") for (int j = 0; j < NLD; ++j) __builtin_amdgcn_global_load_lds((const unsigned*)(base + voff[j]), (LAS unsigned*)(lds + lslot + ldso + j * 1024), 16, 0, 0); \
        lslot = (lslot + STAGE_BYTES == S * STAGE_BYTES) ? 0u : lslot + STAGE_BYTES; \
        if (++lk == NKT) { lk = 0; if (++lu == 1) { ubase = unit_base(1); rot = (idx & 3) * (KB1_ / 4) + (idx >> 2); kbm = KB1_ - 1; } } \
        else if (lk == NKT / 2 && gsrc >= 0) wait_x(1); } while (0)
#define WAITN(n) asm volatile("s_waitcnt vmcnt(%0)" :: "n"((n) * NLD) : "memory")
#pragma unroll
        for (int i = 0; i < S - 1; ++i) ISSUE();
        WAITN(S - 3);
        bar();
#pragma unroll 1
        for (int g = 0; g < G; ++g) {
            if (g + S - 1 < G) { ISSUE(); WAITN(S - 3); }
            else asm volatile("s_waitcnt vmcnt(0)" ::: "memory");
            bar();
        }
    }
}
}

extern "C" void kernel_launch(int rank, int nranks, ncclComm_t comm, hipStream_t stream,
                              void* const* d_in, const int* in_sizes, int n_in,
                              void* d_out, int out_size, const int* in_dtypes, int out_dtype,
                              void* const* d_out_all, void* const* d_ws_all, size_t ws_bytes) {
    if (nranks != NR || (unsigned)rank >= (unsigned)NR || ws_bytes < WS_XSTAGE + 2 * NR * XBLK_BYTES) { fprintf(stderr, "unexpected configuration\n"); return; }
    static bool init[16] = {false};
    int dev = 0; HIP_CHECK(hipGetDevice(&dev));
    if (!init[dev & 15]) {
        HIP_CHECK(hipFuncSetAttribute((const void*)fused_gemm_a2a, hipFuncAttributeMaxDynamicSharedMemorySize, LDS_BYTES));
        init[dev & 15] = true;
    }
    Params p;
    p.x = (const unsigned short*)d_in[0]; p.w = (const unsigned short*)d_in[1];
    for (int i = 0; i < 8; ++i) { p.out_all[i] = d_out_all[i]; p.ws_all[i] = d_ws_all[i]; }
    p.rank = rank;
    p.single = 0;
    hipLaunchKernelGGL(fused_gemm_a2a, dim3(NWG), dim3(NTHREADS), LDS_BYTES, stream, p);
    HIP_CHECK(hipGetLastError());
}
```
